```python
import numpy as np

kernel_name = "gpu_gemm_rs_m4096_k4096_n16384_e4m3e5m2_dqrelu"


ABSMAX_RTOL = 0.05

COMPUTE_DTYPE = "float32"
EPILOGUE = "dequant_relu"
IN_QUANT = None
OUT_QUANT = None
IN_FP8 = "e4m3e5m2"
IN_DTYPES = {"x": "e4m3", "w_mat": "e5m2",
             "scale_x": "float32", "scale_w": "float32"}
WORLD_SIZE = 8

def _snap_fp8(x, mant_bits, max_abs, min_normal):
    a = np.abs(x.astype(np.float64))
    _m, _e = np.frexp(a)
    q = np.where(a >= min_normal, np.ldexp(1.0, _e - 1 - mant_bits),
                 min_normal * 2.0 ** -mant_bits)
    snapped = np.minimum(np.round(a / q) * q, max_abs)
    return (np.sign(x) * snapped).astype(np.float32)

def setup_inputs():

    x = _snap_fp8(np.random.randn(4096, 4096).astype(np.float32), 3, 448.0, 2.0 ** -6)
    w_mat = _snap_fp8(np.random.randn(4096, 16384).astype(np.float32), 2, 57344.0, 2.0 ** -14)
    scale_x = (0.02 * (1.0 + np.random.rand(1))).astype(np.float32)
    scale_w = (0.02 * (1.0 + np.random.rand(1))).astype(np.float32)
    return {"x": x, "w_mat": w_mat, "scale_x": scale_x, "scale_w": scale_w}

def reference(x, w_mat, scale_x, scale_w):
    acc = (x @ w_mat).astype(np.float32)
    y = (acc * (scale_x * scale_w)).astype(np.float32)
    return np.maximum(y, 0.0).astype(np.float32)
```

```cpp
#include <hip/hip_runtime.h>
#include <rccl/rccl.h>
#include <cstdio>
#include <cstdint>

#define HIP_CHECK(expr) do { hipError_t _e = (expr); if (_e != hipSuccess) { \
    fprintf(stderr, "HIP error %s at %s:%d\n", hipGetErrorString(_e), __FILE__, __LINE__); \
    return; } } while (0)

namespace {

constexpr int NR = 8;
constexpr int MG = 4096, KG = 4096, NG = 16384;
constexpr int KS = KG / NR;
constexpr int MS = MG / NR;
constexpr int NS = NG / NR;
constexpr int BM = 128, BN = 256, BK = 128;
constexpr int NCHUNK = MS / BM;
constexpr int NCT = NS / BN;
constexpr int NCOMM = 56, NWG = 256, NGEMM = NWG - NCOMM;
constexpr int GPX = NGEMM / 8;
constexpr int TILES_PER_GROUP = NCHUNK * NR;

constexpr size_t MiB = 1ull << 20;
constexpr size_t OFF_FW  = 0;
constexpr size_t OFF_FX  = 1024;
constexpr size_t OFF_FO  = 4096;
constexpr size_t OFF_TMO = 16384;
constexpr size_t OFF_CNT = 32768;
constexpr size_t OFF_X   = 1 * MiB;
constexpr size_t X_SLAB  = 2 * MiB;
constexpr size_t OFF_W   = OFF_X + 2 * NR * X_SLAB;
constexpr size_t W_SLAB  = 1 * MiB;
constexpr size_t OFF_O   = OFF_W + 2 * NR * W_SLAB;
constexpr size_t O_SLAB  = 4 * MiB;
constexpr size_t WS_USED = OFF_O + NR * O_SLAB;

constexpr unsigned long long DEADLINE_TICKS = 150000000ull;

constexpr int STAGE_BYTES = BM * BK + BK * BN;
constexpr int NSTAGE = 3;
constexpr int LDS_CTRL = NSTAGE * STAGE_BYTES;
constexpr int LDS_BYTES = LDS_CTRL + 64;

typedef unsigned u32x4 __attribute__((ext_vector_type(4)));
typedef float f32x4 __attribute__((ext_vector_type(4)));
typedef int i32x2 __attribute__((ext_vector_type(2)));
typedef int i32x4 __attribute__((ext_vector_type(4)));
typedef int i32x8 __attribute__((ext_vector_type(8)));
#define LDS_AS __attribute__((address_space(3)))

struct Params {
    const unsigned char* x;
    const unsigned char* w;
    const float* sx;
    const float* sw;
    float* out;
    unsigned char* ws[NR];
    int rank;
};

constexpr int AUX_SYS = 17;

__device__ __forceinline__ __amdgpu_buffer_rsrc_t make_rsrc(const void* base, unsigned bytes) {
    return __builtin_amdgcn_make_buffer_rsrc(const_cast<void*>(base), 0, bytes, 0x00020000);
}
__device__ __forceinline__ unsigned ld_sys(const unsigned* p) {
    return __hip_atomic_load(p, __ATOMIC_RELAXED, __HIP_MEMORY_SCOPE_SYSTEM);
}
__device__ __forceinline__ void st_sys(unsigned* p, unsigned v) {
    __hip_atomic_store(p, v, __ATOMIC_RELAXED, __HIP_MEMORY_SCOPE_SYSTEM);
}
#define WAIT_VM0() asm volatile("s_waitcnt vmcnt(0)" ::: "memory")

__device__ __forceinline__ void wait_flags(const unsigned* fp, unsigned call, unsigned* tmo, int sleep_long) {
    const unsigned long long t0 = __builtin_amdgcn_s_memrealtime();
    for (;;) {
        const unsigned v = fp ? ld_sys(fp) : call;
        if (__all((int)(v - call) >= 0)) break;
        const unsigned t = __hip_atomic_load(tmo, __ATOMIC_RELAXED, __HIP_MEMORY_SCOPE_AGENT);
        if (t == call || __builtin_amdgcn_s_memrealtime() - t0 > DEADLINE_TICKS) {
            __hip_atomic_store(tmo, call, __ATOMIC_RELAXED, __HIP_MEMORY_SCOPE_AGENT);
            break;
        }
        if (sleep_long) __builtin_amdgcn_s_sleep(16); else __builtin_amdgcn_s_sleep(2);
    }
}

__device__ __forceinline__ int sigma_x(int r) { return ((r >> 1) & 1) | (((r >> 3) & 1) * 6); }

__global__ __launch_bounds__(512, 2) void gemm_rs_fused(Params p) {
    __shared__ __attribute__((aligned(16))) unsigned char smem[LDS_BYTES];
    const int tid = threadIdx.x;
    const int lane = tid & 63;
    const int wave = __builtin_amdgcn_readfirstlane(tid >> 6);
    const int rank = p.rank;
    unsigned char* const wsme = p.ws[rank];
    unsigned* const tmo = (unsigned*)(wsme + OFF_TMO);
    unsigned* const cntp = (unsigned*)(wsme + OFF_CNT + (size_t)blockIdx.x * 64);

    const unsigned call = __builtin_amdgcn_readfirstlane(
        __hip_atomic_load(cntp, __ATOMIC_RELAXED, __HIP_MEMORY_SCOPE_AGENT)) + 1u;
    const unsigned par = call & 1u;
    unsigned* const ctrl = (unsigned*)(smem + LDS_CTRL);
    if (tid < 16) ctrl[tid] = 0u;
    __syncthreads();
    if (tid == 0) __hip_atomic_store(cntp, call, __ATOMIC_RELAXED, __HIP_MEMORY_SCOPE_AGENT);

    if (blockIdx.x < NCOMM) {
        const int s = blockIdx.x & 7;
        const unsigned q = (unsigned)(rank + 1 + (int)(blockIdx.x >> 3)) & 7u;
        unsigned char* const wsq = p.ws[q];
        const __amdgpu_buffer_rsrc_t rw = make_rsrc(p.w, (unsigned)((size_t)KS * NG));
        const __amdgpu_buffer_rsrc_t rx = make_rsrc(p.x, (unsigned)((size_t)MG * KS));
        const __amdgpu_buffer_rsrc_t rq = make_rsrc(wsq, (unsigned)WS_USED);

        const unsigned w_src0 = (unsigned)(s * 64 + (tid >> 7)) * NG + q * NS + (tid & 127) * 16;
        const unsigned w_dst0 = (unsigned)(OFF_W + (par * NR + rank) * W_SLAB) + s * 131072 + tid * 16;
        const unsigned x_src0 = (unsigned)s * 262144 + tid * 16;
        const unsigned x_dst0 = (unsigned)(OFF_X + (par * NR + rank) * X_SLAB) + x_src0;
#pragma unroll
        for (int b = 0; b < 12; ++b) {
            u32x4 v[4];
#pragma unroll
            for (int u = 0; u < 4; ++u) {
                const int t = b * 4 + u;
                if (t < 16) v[u] = __builtin_amdgcn_raw_buffer_load_b128(rw, w_src0 + t * 4 * NG, 0, 0);
                else        v[u] = __builtin_amdgcn_raw_buffer_load_b128(rx, x_src0 + (t - 16) * 8192, 0, 0);
            }
            WAIT_VM0();
            if (b >= 4 && (b & 1) == 0) {
                const int pc = (b - 4) >> 1;
                if (lane == 0) {
                    const unsigned old = __hip_atomic_fetch_add(&ctrl[pc], 1u, __ATOMIC_RELAXED, __HIP_MEMORY_SCOPE_WORKGROUP);
                    if (old == 7u) {
                        unsigned* f = pc == 0 ? (unsigned*)(wsq + OFF_FW) + rank * 8 + s
                                              : (unsigned*)(wsq + OFF_FX) + ((pc - 1) * 8 + s) * 8 + rank;
                        st_sys(f, call);
                    }
                }
            }
#pragma unroll
            for (int u = 0; u < 4; ++u) {
                const int t = b * 4 + u;
                if (t < 16) __builtin_amdgcn_raw_buffer_store_b128(v[u], rq, w_dst0 + t * 8192, 0, AUX_SYS);
                else        __builtin_amdgcn_raw_buffer_store_b128(v[u], rq, x_dst0 + (t - 16) * 8192, 0, AUX_SYS);
            }
        }
        WAIT_VM0();
        if (lane == 0) {
            const unsigned old = __hip_atomic_fetch_add(&ctrl[4], 1u, __ATOMIC_RELAXED, __HIP_MEMORY_SCOPE_WORKGROUP);
            if (old == 7u) st_sys((unsigned*)(wsq + OFF_FX) + (3 * 8 + s) * 8 + rank, call);
        }

        const __amdgpu_buffer_rsrc_t ro = make_rsrc(p.out, (unsigned)((size_t)MS * NG * 4));
        const int wm = wave >> 2, wn = wave & 3;
        for (int j = 0; j < NCHUNK; ++j) {
            const unsigned* f = (const unsigned*)(wsme + OFF_FO) + ((q * 4 + j) * 8 + s) * 8 + wave;
            wait_flags(lane == 0 ? f : nullptr, call, tmo, 0);
            const bool bad = __hip_atomic_load(tmo, __ATOMIC_RELAXED, __HIP_MEMORY_SCOPE_AGENT) == call;
            asm volatile("" ::: "memory");
            const unsigned src0 = (unsigned)(OFF_O + (size_t)rank * O_SLAB) + ((j * 8 + s) * 8 + wave) * 16384 + lane * 16;
            const unsigned dst0 = ((unsigned)(j * BM + wm * 64 + (lane & 15)) * NG + q * NS + s * BN + wn * 64 + (lane >> 4) * 4) * 4;
#pragma unroll
            for (int h = 0; h < 16; h += 2) {
                u32x4 v[2];
#pragma unroll
                for (int u = 0; u < 2; ++u) {
                    if (!bad) v[u] = __builtin_amdgcn_raw_buffer_load_b128(rq, src0 + (h + u) * 1024, 0, AUX_SYS);
                    else v[u] = u32x4{0x7fc00000u, 0x7fc00000u, 0x7fc00000u, 0x7fc00000u};
                }
#pragma unroll
                for (int u = 0; u < 2; ++u) {
                    const int fr = h + u, ni = fr >> 2, mi = fr & 3;
                    __builtin_amdgcn_raw_buffer_store_b128(v[u], ro, dst0 + (mi * 16 * NG + ni * 16) * 4, 0, 0);
                }
            }
        }
    } else {
        const int gi = blockIdx.x - NCOMM;
        const int ct = gi & 7;
        const int wm = wave >> 2, wn = wave & 3;
        const float scale = p.sx[0] * p.sw[0];
        const unsigned char* const wsX = wsme + OFF_X + (size_t)par * NR * X_SLAB;
        const unsigned char* const wsW = wsme + OFF_W + (size_t)par * NR * W_SLAB;

        unsigned xoff[2];
#pragma unroll
        for (int i = 0; i < 2; ++i) {
            const int r = wave * 16 + i * 8 + (lane >> 3);
            xoff[i] = r * KS + (((lane & 7) ^ sigma_x(r & 15)) << 4);
        }
        unsigned wrow[4], wch[4];
#pragma unroll
        for (int i = 0; i < 4; ++i) {
            const int kr = wave * 16 + i * 4 + (lane >> 4);
            wrow[i] = kr;
            wch[i] = (((lane & 15) ^ ((kr & 7) | ((kr >> 2) & 8))) << 4);
        }
        const int g = lane >> 4, t16 = lane & 15;
        unsigned xa[2];
#pragma unroll
        for (int h = 0; h < 2; ++h) xa[h] = (wm * 64 + t16) * BK + (((2 * g + h) ^ sigma_x(t16)) << 4);
        unsigned wa[4];
        {
            const int F = (t16 >> 1) | ((g & 1) << 3);
#pragma unroll
            for (int ni = 0; ni < 4; ++ni)
                wa[ni] = BM * BK + (32 * g + (t16 >> 1)) * BN + (((wn * 4 + ni) ^ F) << 4) + (t16 & 1) * 8;
        }
        const unsigned lds_base = (unsigned)(uintptr_t)(LDS_AS unsigned char*)smem;

        for (int idx = gi >> 3; idx < TILES_PER_GROUP; idx += GPX) {
            int j, qq;
            if (idx < 28) { j = idx / 7; qq = (rank + 1 + idx % 7) & 7; }
            else          { j = idx - 28; qq = rank; }
            const int R0 = qq * MS + j * BM;

            if (wave == 0) {
                const unsigned* f = nullptr;
                if (lane < 56) {
                    int src = lane >> 3; src += (src >= rank);
                    f = (const unsigned*)(wsme + OFF_FW) + src * 8 + (lane & 7);
                } else if (lane < 63) {
                    int src = lane - 56; src += (src >= rank);
                    f = (const unsigned*)(wsme + OFF_FX) + (j * 8 + qq) * 8 + src;
                }
                wait_flags(f, call, tmo, 1);
                __builtin_amdgcn_fence(__ATOMIC_ACQUIRE, "");
                WAIT_VM0();
            }
            __syncthreads();

            auto stage = [&](int kt, int buf) {
                const int ps = kt >> 2, kk = kt & 3;
                const bool own = (ps == rank);
                const unsigned char* xb = (own ? p.x : wsX + (size_t)ps * X_SLAB) + (size_t)R0 * KS + kk * BK;
                const unsigned char* wb = (own ? p.w + rank * NS : wsW + (size_t)ps * W_SLAB) + ct * BN;
                const int sh = own ? 14 : 11;
                unsigned char* lb = smem + buf * STAGE_BYTES;
#pragma unroll
                for (int i = 0; i < 2; ++i)
                    __builtin_amdgcn_global_load_lds((const unsigned*)(xb + xoff[i]),
                        (LDS_AS unsigned*)(lb + (wave * 2 + i) * 1024), 16, 0, 0);
#pragma unroll
                for (int i = 0; i < 4; ++i)
                    __builtin_amdgcn_global_load_lds((const unsigned*)(wb + (((kk * BK + wrow[i]) << sh) + wch[i])),
                        (LDS_AS unsigned*)(lb + BM * BK + (wave * 4 + i) * 1024), 16, 0, 0);
            };

            f32x4 acc[4][4];
#pragma unroll
            for (int a = 0; a < 4; ++a)
#pragma unroll
                for (int b = 0; b < 4; ++b) acc[a][b] = f32x4{0.f, 0.f, 0.f, 0.f};

            constexpr int NKT = KG / BK;
            stage(0, 0);
            stage(1, 1);
            int buf = 0;
            for (int kt = 0; kt < NKT; ++kt) {
                if (kt + 1 < NKT) asm volatile("s_waitcnt vmcnt(6)" ::: "memory");
                else WAIT_VM0();
                __builtin_amdgcn_s_barrier();
                asm volatile("" ::: "memory");
                if (kt + 2 < NKT) { int nb = buf + 2; if (nb >= NSTAGE) nb -= NSTAGE; stage(kt + 2, nb); }
                const unsigned lb = lds_base + buf * STAGE_BYTES;
                i32x8 xf0, xf1, xf2, xf3, wf0, wf1, wf2, wf3;
                const unsigned xa0 = lb + xa[0], xa1 = lb + xa[1];
                asm volatile(
                    "ds_read_b64_tr_b8 v[160:161], %5 offset:0\n\t"
                    "ds_read_b64_tr_b8 v[162:163], %5 offset:2048\n\t"
                    "ds_read_b64_tr_b8 v[164:165], %5 offset:4096\n\t"
                    "ds_read_b64_tr_b8 v[166:167], %5 offset:6144\n\t"
                    "ds_read_b128 v[128:131], %6 offset:0\n\t"
                    "ds_read_b128 v[132:135], %7 offset:0\n\t"
                    "ds_read_b128 v[136:139], %6 offset:2048\n\t"
                    "ds_read_b128 v[140:143], %7 offset:2048\n\t"
                    "ds_read_b128 v[144:147], %6 offset:4096\n\t"
                    "ds_read_b128 v[148:151], %7 offset:4096\n\t"
                    "ds_read_b128 v[152:155], %6 offset:6144\n\t"
                    "ds_read_b128 v[156:159], %7 offset:6144"
                    : "={v[160:167]}"(wf0), "={v[128:135]}"(xf0), "={v[136:143]}"(xf1), "={v[144:151]}"(xf2), "={v[152:159]}"(xf3)
                    : "v"(lb + wa[0]), "v"(xa0), "v"(xa1));
                asm volatile(
                    "ds_read_b64_tr_b8 v[168:169], %3 offset:0\n\t"
                    "ds_read_b64_tr_b8 v[170:171], %3 offset:2048\n\t"
                    "ds_read_b64_tr_b8 v[172:173], %3 offset:4096\n\t"
                    "ds_read_b64_tr_b8 v[174:175], %3 offset:6144\n\t"
                    "ds_read_b64_tr_b8 v[176:177], %4 offset:0\n\t"
                    "ds_read_b64_tr_b8 v[178:179], %4 offset:2048\n\t"
                    "ds_read_b64_tr_b8 v[180:181], %4 offset:4096\n\t"
                    "ds_read_b64_tr_b8 v[182:183], %4 offset:6144\n\t"
                    "ds_read_b64_tr_b8 v[184:185], %5 offset:0\n\t"
                    "ds_read_b64_tr_b8 v[186:187], %5 offset:2048\n\t"
                    "ds_read_b64_tr_b8 v[188:189], %5 offset:4096\n\t"
                    "ds_read_b64_tr_b8 v[190:191], %5 offset:6144"
                    : "={v[168:175]}"(wf1), "={v[176:183]}"(wf2), "={v[184:191]}"(wf3)
                    : "v"(lb + wa[1]), "v"(lb + wa[2]), "v"(lb + wa[3]));
#define MFMA4(NI, WF) do { \
                    acc[NI][0] = __builtin_amdgcn_mfma_scale_f32_16x16x128_f8f6f4(WF, xf0, acc[NI][0], 1, 0, 0, 0, 0, 0); \
                    acc[NI][1] = __builtin_amdgcn_mfma_scale_f32_16x16x128_f8f6f4(WF, xf1, acc[NI][1], 1, 0, 0, 0, 0, 0); \
                    acc[NI][2] = __builtin_amdgcn_mfma_scale_f32_16x16x128_f8f6f4(WF, xf2, acc[NI][2], 1, 0, 0, 0, 0, 0); \
                    acc[NI][3] = __builtin_amdgcn_mfma_scale_f32_16x16x128_f8f6f4(WF, xf3, acc[NI][3], 1, 0, 0, 0, 0, 0); \
                    __builtin_amdgcn_sched_barrier(0); } while (0)
                asm volatile("s_waitcnt lgkmcnt(12)"
                    : "+{v[160:167]}"(wf0), "+{v[128:135]}"(xf0), "+{v[136:143]}"(xf1), "+{v[144:151]}"(xf2), "+{v[152:159]}"(xf3));
                MFMA4(0, wf0);
                asm volatile("s_waitcnt lgkmcnt(8)" : "+{v[168:175]}"(wf1));
                MFMA4(1, wf1);
                asm volatile("s_waitcnt lgkmcnt(4)" : "+{v[176:183]}"(wf2));
                MFMA4(2, wf2);
                asm volatile("s_waitcnt lgkmcnt(0)" : "+{v[184:191]}"(wf3));
                MFMA4(3, wf3);
                asm volatile("" ::: "memory");
                buf = (buf + 1 == NSTAGE) ? 0 : buf + 1;
            }

            const bool bad = __hip_atomic_load(tmo, __ATOMIC_RELAXED, __HIP_MEMORY_SCOPE_AGENT) == call;
            const float qnan = __builtin_bit_cast(float, 0x7fc00000u);
            if (qq == rank) {
                const __amdgpu_buffer_rsrc_t ro = make_rsrc(p.out, (unsigned)((size_t)MS * NG * 4));
                const unsigned dst0 = ((unsigned)(j * BM + wm * 64 + t16) * NG + rank * NS + ct * BN + wn * 64 + g * 4) * 4;
#pragma unroll
                for (int ni = 0; ni < 4; ++ni)
#pragma unroll
                    for (int mi = 0; mi < 4; ++mi) {
                        f32x4 y;
#pragma unroll
                        for (int e = 0; e < 4; ++e) { const float v = acc[ni][mi][e] * scale; y[e] = bad ? qnan : (v < 0.f ? 0.f : v); }
                        __builtin_amdgcn_raw_buffer_store_b128(__builtin_bit_cast(u32x4, y), ro, dst0 + (mi * 16 * NG + ni * 16) * 4, 0, 0);
                    }
            } else {
                const __amdgpu_buffer_rsrc_t rs = make_rsrc(wsme, (unsigned)WS_USED);
                const unsigned dst0 = (unsigned)(OFF_O + (size_t)qq * O_SLAB) + ((j * 8 + ct) * 8 + wave) * 16384 + lane * 16;
#pragma unroll
                for (int ni = 0; ni < 4; ++ni)
#pragma unroll
                    for (int mi = 0; mi < 4; ++mi) {
                        f32x4 y;
#pragma unroll
                        for (int e = 0; e < 4; ++e) { const float v = acc[ni][mi][e] * scale; y[e] = bad ? qnan : (v < 0.f ? 0.f : v); }
                        __builtin_amdgcn_raw_buffer_store_b128(__builtin_bit_cast(u32x4, y), rs, dst0 + (ni * 4 + mi) * 1024, 0, AUX_SYS);
                    }
                WAIT_VM0();
                if (lane == 0)
                    st_sys((unsigned*)(p.ws[qq] + OFF_FO) + ((rank * 4 + j) * 8 + ct) * 8 + wave, call);
            }
            __syncthreads();
        }
    }
}

}

extern "C" void kernel_launch(int rank, int nranks, ncclComm_t comm,
                              hipStream_t stream,
                              void* const* d_in, const int* in_sizes, int n_in,
                              void* d_out, int out_size,
                              const int* in_dtypes, int out_dtype,
                              void* const* d_out_all,
                              void* const* d_ws_all, size_t ws_bytes) {
    (void)comm; (void)in_dtypes; (void)out_dtype; (void)d_out_all;
    if (nranks != NR || n_in < 4 || rank < 0 || rank >= NR || ws_bytes < WS_USED ||
        in_sizes[0] != MG * KS || in_sizes[1] != KS * NG || out_size != MS * NG) {
        fprintf(stderr, "kernel_launch: unexpected problem shape\n");
        return;
    }
    Params p;
    p.x = (const unsigned char*)d_in[0];
    p.w = (const unsigned char*)d_in[1];
    p.sx = (const float*)d_in[2];
    p.sw = (const float*)d_in[3];
    p.out = (float*)d_out;
    for (int i = 0; i < NR; ++i) p.ws[i] = (unsigned char*)d_ws_all[i];
    p.rank = rank;
    hipLaunchKernelGGL(gemm_rs_fused, dim3(NWG), dim3(512), 0, stream, p);
    HIP_CHECK(hipGetLastError());
}
```

```cpp
#include <hip/hip_runtime.h>
#include <rccl/rccl.h>
#include <cstdio>
#include <cstdint>

#define HIP_CHECK(expr) do { hipError_t _e = (expr); if (_e != hipSuccess) { \
    fprintf(stderr, "HIP error %s at %s:%d\n", hipGetErrorString(_e), __FILE__, __LINE__); \
    return; } } while (0)

namespace {

constexpr int NR = 8;
constexpr int MG = 4096, KG = 4096, NG = 16384;
constexpr int KS = KG / NR;
constexpr int MS = MG / NR;
constexpr int NS = NG / NR;
constexpr int BM = 128, BN = 128, BK = 128;
constexpr int NCHUNK = MS / BM;
constexpr int NCT = NS / BN;
constexpr int NCOMM = 56, NWG = 256;
#define GPX 25
#define PULL_DEPTH 2
#define OUT_AUX 16
#define NSTAGE 4
constexpr int TILES_PER_GROUP = NCHUNK * NR * 2;
constexpr int REMOTE_TILES = NCHUNK * (NR - 1) * 2;

constexpr unsigned U_W = 256;
constexpr unsigned U_XP = 16;
constexpr unsigned U_XEND = U_W + NCHUNK * NR * U_XP;
constexpr unsigned U_PULL = NCHUNK * NCT * 8 * 2;
constexpr unsigned U_ALL = U_XEND + U_PULL;
constexpr unsigned LINK_WAVES = 64;

constexpr size_t MiB = 1ull << 20;
constexpr size_t OFF_FW  = 0;
constexpr size_t OFF_FD  = 256;
constexpr size_t OFF_FS  = 512;
constexpr size_t OFF_FX  = 1536;
constexpr size_t OFF_TK  = 3072;
constexpr size_t OFF_FO  = 4096;
constexpr size_t OFF_TMO = 24576;
constexpr size_t OFF_CNT = 32768;
constexpr size_t OFF_PC  = 65536;
constexpr size_t X_SLAB  = 2 * MiB;
constexpr size_t OFF_XS  = 1 * MiB;
constexpr size_t OFF_X   = OFF_XS + 2 * X_SLAB;
constexpr size_t OFF_W   = OFF_X + NR * X_SLAB;
constexpr size_t W_SLAB  = 1 * MiB;
constexpr size_t OFF_O   = OFF_W + 2 * NR * W_SLAB;
constexpr size_t O_SLAB  = 4 * MiB;
constexpr size_t WS_USED = OFF_O + NR * O_SLAB;

constexpr unsigned long long DEADLINE_TICKS = 150000000ull;

constexpr int STAGE_BYTES = BM * BK + BK * BN;
constexpr int LDS_BYTES = NSTAGE * STAGE_BYTES;

typedef unsigned u32x4 __attribute__((ext_vector_type(4)));
typedef float f32x4 __attribute__((ext_vector_type(4)));
typedef int i32x8 __attribute__((ext_vector_type(8)));
#define LDS_AS __attribute__((address_space(3)))

struct Params {
    const unsigned char* x;
    const unsigned char* w;
    const float* sx;
    const float* sw;
    float* out;
    unsigned char* ws[NR];
    int rank;
};

constexpr int AUX_AGENT = 16;
constexpr int AUX_SYS = 17;

__device__ __forceinline__ __amdgpu_buffer_rsrc_t make_rsrc(const void* base, unsigned bytes) {
    return __builtin_amdgcn_make_buffer_rsrc(const_cast<void*>(base), 0, bytes, 0x00020000);
}
__device__ __forceinline__ unsigned ld_sys(const unsigned* p) {
    return __hip_atomic_load(p, __ATOMIC_RELAXED, __HIP_MEMORY_SCOPE_SYSTEM);
}
__device__ __forceinline__ void st_sys(unsigned* p, unsigned v) {
    __hip_atomic_store(p, v, __ATOMIC_RELAXED, __HIP_MEMORY_SCOPE_SYSTEM);
}
#define WAIT_VM0() asm volatile("s_waitcnt vmcnt(0)" ::: "memory")
#define DRAW_ASYNC(REG, ctr) do { if (lane == 0) \
    asm volatile("global_atomic_add " REG ", %0, %1, off sc0" :: "v"(ctr), "v"(1u) : REG, "memory"); } while (0)
#define STR_(x) #x
#define STR(x) STR_(x)
#define DRAW_GET(REG, N, dst) \
    asm volatile("s_waitcnt vmcnt(" STR(N) ")\n\tv_readfirstlane_b32 %0, " REG "\n\ts_nop 3" : "=s"(dst) :: REG, "memory")
#define STAMP(i) do { } while (0)
#define STAMPV(i, v) do { } while (0)

__device__ __forceinline__ bool wait_flags(const unsigned* fp, unsigned want, unsigned call, unsigned* tmo, int sleep_long) {
    const unsigned long long t0 = __builtin_amdgcn_s_memrealtime();
    for (;;) {
        const unsigned v = fp ? ld_sys(fp) : want;
        const unsigned t = __hip_atomic_load(tmo, __ATOMIC_RELAXED, __HIP_MEMORY_SCOPE_AGENT);
        if (t == call) return false;
        if (__all((int)(v - want) >= 0)) return true;
        if (__builtin_amdgcn_s_memrealtime() - t0 > DEADLINE_TICKS) {
            __hip_atomic_store(tmo, call, __ATOMIC_RELAXED, __HIP_MEMORY_SCOPE_AGENT);
            return false;
        }
        if (sleep_long) __builtin_amdgcn_s_sleep(16); else __builtin_amdgcn_s_sleep(2);
    }
}

__device__ __forceinline__ int sigma_x(int r) { return ((r >> 1) & 1) | (((r >> 3) & 1) * 6); }

__global__ __launch_bounds__(512, 2) void gemm_rs_fused(Params p) {
    __shared__ __attribute__((aligned(16))) unsigned char smem[LDS_BYTES];
    const int tid = threadIdx.x;
    const int lane = tid & 63;
    const int wave = __builtin_amdgcn_readfirstlane(tid >> 6);
    const int rank = p.rank;
    unsigned char* const wsme = p.ws[rank];
    unsigned* const tmo = (unsigned*)(wsme + OFF_TMO);
    unsigned* const cntp = (unsigned*)(wsme + OFF_CNT + (size_t)blockIdx.x * 64);

    const unsigned cnt_raw = __hip_atomic_load(cntp, __ATOMIC_RELAXED, __HIP_MEMORY_SCOPE_AGENT);

    if (blockIdx.x < NCOMM) {
        const int li = blockIdx.x >> 3;
        const unsigned q = (unsigned)(rank + 1 + li) & 7u;
        unsigned char* const wsq = p.ws[q];
        unsigned* const tkp = (unsigned*)(wsme + OFF_TK + li * 128);
        const __amdgpu_buffer_rsrc_t rw = make_rsrc(p.w, (unsigned)((size_t)KS * NG));
        const __amdgpu_buffer_rsrc_t rq = make_rsrc(wsq, (unsigned)WS_USED);
        const __amdgpu_buffer_rsrc_t rm = make_rsrc(wsme, (unsigned)WS_USED);
        const __amdgpu_buffer_rsrc_t ro = make_rsrc(p.out, (unsigned)((size_t)MS * NG * 4));

        unsigned u = (blockIdx.x & 7) * 8 + wave;
        DRAW_ASYNC("v200", tkp);
        const unsigned l16 = lane * 16;
        u32x4 v[4];
#pragma unroll
        for (int i = 0; i < 4; ++i)
            v[i] = __builtin_amdgcn_raw_buffer_load_b128(rw, l16 + (i & 1) * 1024, (2 * u + (i >> 1)) * NG + q * NS, 0);
        const unsigned call = __builtin_amdgcn_readfirstlane(cnt_raw) + 1u;
        const unsigned par = call & 1u;
        const unsigned tk_base = (call - 1u) * U_ALL - LINK_WAVES;
        __syncthreads();
        if (tid == 0) __hip_atomic_store(cntp, call, __ATOMIC_RELAXED, __HIP_MEMORY_SCOPE_AGENT);
        STAMP(0);
        wait_flags(lane == 0 ? (const unsigned*)(wsme + OFF_FD) + q : nullptr, call - 1u, call, tmo, 0);

        const unsigned w_dst = (unsigned)(OFF_W + (par * NR + rank) * W_SLAB);
        unsigned* const pcb = (unsigned*)(wsme + OFF_PC + li * 4096);
        auto publish = [&](unsigned pc, unsigned old) {
            if (lane == 0) {
                if (pc == 0u) {
                    if (old + 1u == call * U_W) st_sys((unsigned*)(wsq + OFF_FW) + rank, call);
                } else if (old + 1u == call * U_XP) {
                    __hip_atomic_store((unsigned*)(wsme + OFF_FX) + (pc - 1u) * 8 + q, call, __ATOMIC_RELAXED, __HIP_MEMORY_SCOPE_AGENT);
                }
            }
        };
        unsigned pc_prev = ~0u;
        for (;;) {
            unsigned tn, old = 0;
            asm volatile("" : "+v"(v[0]), "+v"(v[1]), "+v"(v[2]), "+v"(v[3]));
            DRAW_GET("v200", 0, tn);
            if (pc_prev != ~0u) DRAW_ASYNC("v201", pcb);
#pragma unroll
            for (int i = 0; i < 4; ++i) __builtin_amdgcn_raw_buffer_store_b128(v[i], rq, l16 + i * 1024, w_dst + u * 4096, AUX_SYS);
            u = tn - tk_base;
            if (u < U_ALL) DRAW_ASYNC("v200", tkp);
            if (u < U_W) {
#pragma unroll
                for (int i = 0; i < 4; ++i)
                    v[i] = __builtin_amdgcn_raw_buffer_load_b128(rw, l16 + (i & 1) * 1024, (2 * u + (i >> 1)) * NG + q * NS, 0);
                if (pc_prev != ~0u) {
                    DRAW_GET("v201", 9, old);
                    publish(0, old);
                }
                pc_prev = 0;
            } else {
                if (pc_prev != ~0u) { DRAW_GET("v201", 4, old); publish(0, old); }
                pc_prev = 0;
                break;
            }
        }

        if (u < U_XEND)
            wait_flags(lane < 32 ? (const unsigned*)(wsme + OFF_FS) + q * 32 + lane : nullptr, call, call, tmo, 0);
        while (u < U_XEND) {
            const unsigned t = u - U_W, pc = 1 + (t >> 4);
            const unsigned off = ((t >> 4 & 7) * MS + (t >> 7) * BM) * KS + (t & 15) * 4096;
            const unsigned src0 = (unsigned)(OFF_XS + par * X_SLAB) + off;
            const unsigned dst0 = (unsigned)(OFF_X + q * X_SLAB) + off;
            unsigned un = 0, old;
#pragma unroll
            for (int h = 0; h < 4; h += PULL_DEPTH) {
#pragma unroll
                for (int f = h; f < h + PULL_DEPTH; ++f)
                    v[f] = __builtin_amdgcn_raw_buffer_load_b128(rq, l16 + f * 1024, src0, AUX_SYS);
                if (h == 0) {
                    DRAW_GET("v200", PULL_DEPTH, un);
                    un -= tk_base;
                    if (un < U_ALL) DRAW_ASYNC("v200", tkp);
                    DRAW_ASYNC("v201", pcb + pc_prev * 16);
                }
#pragma unroll
                for (int f = h; f < h + PULL_DEPTH; ++f)
                    __builtin_amdgcn_raw_buffer_store_b128(v[f], rm, l16 + f * 1024, dst0, AUX_AGENT);
                if (h == 0) {
                    DRAW_GET("v201", PULL_DEPTH, old);
                    publish(pc_prev, old);
                }
            }
            pc_prev = pc;
            u = un;
            if (u >= U_XEND) break;
        }
        {
            unsigned old;
            WAIT_VM0();
            DRAW_ASYNC("v201", pcb + pc_prev * 16);
            DRAW_GET("v201", 0, old);
            publish(pc_prev, old);
        }
        STAMP(6);

        const unsigned olane = ((lane & 15) * NG + (lane >> 4) * 4) * 4;
        const unsigned* const fo = (const unsigned*)(wsme + OFF_FO) + q * 512;
        bool ready = false;
        while (u < U_ALL) {
            const unsigned t = u - U_XEND;
            const unsigned blk = t >> 1, hh = t & 1;
            const unsigned j = blk >> 7, ct = (blk >> 3) & 15, w = blk & 7;
            bool ok = true;
            if (!ready) {
                ok = wait_flags(lane == 0 ? fo + blk : nullptr, call, call, tmo, 0);
                asm volatile("" ::: "memory");
            }
            const unsigned src0 = (unsigned)(OFF_O + (size_t)rank * O_SLAB) + blk * 8192 + hh * 2048;
            const unsigned dst0 = ((j * BM + (w >> 2) * 64 + hh * 32) * NG + q * NS + ct * BN + (w & 3) * 32) * 4;
            unsigned un = 0;
            if (ok) {
#pragma unroll
                for (int h = 0; h < 4; h += PULL_DEPTH) {
#pragma unroll
                    for (int f = h; f < h + PULL_DEPTH; ++f)
                        v[f] = __builtin_amdgcn_raw_buffer_load_b128(rq, l16 + (f >> 1) * 1024, src0 + (f & 1) * 4096, AUX_SYS);
                    if (h == 0) {
                        DRAW_GET("v200", PULL_DEPTH, un);
                        un -= tk_base;
                        if (un < U_ALL) {
                            DRAW_ASYNC("v200", tkp);
                            asm volatile("global_load_dword v202, %0, off sc0 sc1" :: "v"(fo + ((un - U_XEND) >> 1)) : "v202", "memory");
                        }
                    }
#pragma unroll
                    for (int f = h; f < h + PULL_DEPTH; ++f)
                        __builtin_amdgcn_raw_buffer_store_b128(v[f], ro, olane + (f & 1) * 64, dst0 + (f >> 1) * 16 * NG * 4, OUT_AUX);
                }
                ready = false;
                if (un < U_ALL) {
                    unsigned fv;
                    DRAW_GET("v202", 4, fv);
                    ready = (int)(fv - call) >= 0;
                }
            } else {
#pragma unroll
                for (int f = 0; f < 4; ++f)
                    __builtin_amdgcn_raw_buffer_store_b128(u32x4{0x7fc00000u, 0x7fc00000u, 0x7fc00000u, 0x7fc00000u}, ro,
                                                           olane + (f & 1) * 64, dst0 + (f >> 1) * 16 * NG * 4, OUT_AUX);
                DRAW_GET("v200", 0, un);
                un -= tk_base;
                if (un < U_ALL) DRAW_ASYNC("v200", tkp);
                ready = false;
            }
            u = un;
        }
        {
            unsigned old;
            WAIT_VM0();
            DRAW_ASYNC("v201", pcb + 33 * 16);
            DRAW_GET("v201", 0, old);
            if (lane == 0 && old + 1u == call * LINK_WAVES) st_sys((unsigned*)(wsq + OFF_FD) + rank, call);
        }
        STAMP(14);
    } else {
        const unsigned call = __builtin_amdgcn_readfirstlane(cnt_raw) + 1u;
        const unsigned par = call & 1u;
        __syncthreads();
        if (tid == 0) __hip_atomic_store(cntp, call, __ATOMIC_RELAXED, __HIP_MEMORY_SCOPE_AGENT);
        STAMP(0);
        const int gi = blockIdx.x - NCOMM;
        const int grp = gi & 7;
        if (gi < NCHUNK * NR) {
            const unsigned off = ((gi & 7) * MS + (gi >> 3) * BM) * KS + tid * 16;
            const __amdgpu_buffer_rsrc_t rx = make_rsrc(p.x, (unsigned)((size_t)MG * KS));
            const __amdgpu_buffer_rsrc_t rm = make_rsrc(wsme, (unsigned)WS_USED);
            u32x4 v[8];
#pragma unroll
            for (int i = 0; i < 8; ++i) v[i] = __builtin_amdgcn_raw_buffer_load_b128(rx, off + i * 8192, 0, 0);
#pragma unroll
            for (int i = 0; i < 8; ++i)
                __builtin_amdgcn_raw_buffer_store_b128(v[i], rm, off + i * 8192, (unsigned)(OFF_XS + par * X_SLAB), AUX_SYS);
            WAIT_VM0();
            __syncthreads();
            if (tid < NR && tid != rank) st_sys((unsigned*)(p.ws[tid] + OFF_FS) + rank * 32 + gi, call);
        }
        if ((gi >> 3) >= GPX) return;
        const int wm = wave >> 2, wn = wave & 3;
        const float scale = p.sx[0] * p.sw[0];
        const unsigned char* const wsX = wsme + OFF_X;
        const unsigned char* const wsW = wsme + OFF_W + (size_t)par * NR * W_SLAB;

        unsigned xoff[2], wrow[2], wch[2];
#pragma unroll
        for (int i = 0; i < 2; ++i) {
            const int r = wave * 16 + i * 8 + (lane >> 3);
            xoff[i] = r * KS + (((lane & 7) ^ sigma_x(r & 15)) << 4);
            wrow[i] = r;
            wch[i] = (((lane & 7) ^ (((r >> 1) & 3) | (((r >> 5) & 1) << 2))) << 4);
        }
        const int g = lane >> 4, t16 = lane & 15;
        unsigned xa[2], wa[2];
#pragma unroll
        for (int h = 0; h < 2; ++h) xa[h] = (wm * 64 + t16) * BK + (((2 * g + h) ^ sigma_x(t16)) << 4);
        {
            const int F = ((t16 >> 2) & 3) | ((g & 1) << 2);
#pragma unroll
            for (int ni = 0; ni < 2; ++ni)
                wa[ni] = BM * BK + (32 * g + (t16 >> 1)) * BN + (((wn * 2 + ni) ^ F) << 4) + (t16 & 1) * 8;
        }
        const unsigned lds_base = (unsigned)(uintptr_t)(LDS_AS unsigned char*)smem;
        int round = 0;

        for (int idx = gi >> 3; idx < TILES_PER_GROUP; idx += GPX, ++round) {
            int j, qq;
            const int ct = 2 * grp + (idx & 1);
            if (idx < REMOTE_TILES) { const int u = idx >> 1; j = u / 7; qq = (rank + 1 + u % 7) & 7; }
            else                    { j = (idx - REMOTE_TILES) >> 1; qq = rank; }
            const int R0 = qq * MS + j * BM;

            if (wave == 0) {
                const unsigned* f = nullptr;
                const int l7 = lane & 7;
                if (l7 < 7 && lane < 16) {
                    const int src = l7 + (l7 >= rank);
                    if (lane < 8) f = (const unsigned*)(wsme + OFF_FW) + src;
                    else          f = (const unsigned*)(wsme + OFF_FX) + (j * 8 + qq) * 8 + src;
                }
                wait_flags(f, call, call, tmo, 1);
                __builtin_amdgcn_fence(__ATOMIC_ACQUIRE, "");
                WAIT_VM0();
            }
            __syncthreads();
            STAMP(1 + 4 * round);

            auto stage = [&](int kt, int buf) {
                const int ps = kt >> 2, kk = kt & 3;
                const bool own = (ps == rank);
                const unsigned char* xb = (own ? p.x : wsX + (size_t)ps * X_SLAB) + (size_t)R0 * KS + kk * BK;
                const unsigned char* wb = (own ? p.w + rank * NS : wsW + (size_t)ps * W_SLAB) + ct * BN;
                const int sh = own ? 14 : 11;
                unsigned char* lb = smem + buf * STAGE_BYTES;
#pragma unroll
                for (int i = 0; i < 2; ++i)
                    __builtin_amdgcn_global_load_lds((const unsigned*)(xb + xoff[i]),
                        (LDS_AS unsigned*)(lb + (wave * 2 + i) * 1024), 16, 0, 0);
#pragma unroll
                for (int i = 0; i < 2; ++i)
                    __builtin_amdgcn_global_load_lds((const unsigned*)(wb + (((kk * BK + wrow[i]) << sh) + wch[i])),
                        (LDS_AS unsigned*)(lb + BM * BK + (wave * 2 + i) * 1024), 16, 0, 0);
            };

            f32x4 acc[2][4];
#pragma unroll
            for (int a = 0; a < 2; ++a)
#pragma unroll
                for (int b = 0; b < 4; ++b) acc[a][b] = f32x4{0.f, 0.f, 0.f, 0.f};

            constexpr int NKT = KG / BK;
#pragma unroll
            for (int i = 0; i < NSTAGE - 1; ++i) stage(i, i);
            int buf = 0;
            for (int kt = 0; kt < NKT; ++kt) {
                const int younger = (NKT - 1 - kt) < (NSTAGE - 2) ? (NKT - 1 - kt) : (NSTAGE - 2);
                if (younger >= 3) asm volatile("s_waitcnt vmcnt(12)" ::: "memory");
                else if (younger == 2) asm volatile("s_waitcnt vmcnt(8)" ::: "memory");
                else if (younger == 1) asm volatile("s_waitcnt vmcnt(4)" ::: "memory");
                else WAIT_VM0();
                __builtin_amdgcn_s_barrier();
                asm volatile("" ::: "memory");
                if (kt + NSTAGE - 1 < NKT) { int nb = buf + NSTAGE - 1; if (nb >= NSTAGE) nb -= NSTAGE; stage(kt + NSTAGE - 1, nb); }
                const unsigned lb = lds_base + buf * STAGE_BYTES;
                i32x8 xf0, xf1, xf2, xf3, wf0, wf1;
                const unsigned xa0 = lb + xa[0], xa1 = lb + xa[1];
                asm volatile(
                    "ds_read_b64_tr_b8 v[160:161], %5 offset:0\n\t"
                    "ds_read_b64_tr_b8 v[162:163], %5 offset:1024\n\t"
                    "ds_read_b64_tr_b8 v[164:165], %5 offset:2048\n\t"
                    "ds_read_b64_tr_b8 v[166:167], %5 offset:3072\n\t"
                    "ds_read_b128 v[128:131], %6 offset:0\n\t"
                    "ds_read_b128 v[132:135], %7 offset:0\n\t"
                    "ds_read_b128 v[136:139], %6 offset:2048\n\t"
                    "ds_read_b128 v[140:143], %7 offset:2048\n\t"
                    "ds_read_b128 v[144:147], %6 offset:4096\n\t"
                    "ds_read_b128 v[148:151], %7 offset:4096\n\t"
                    "ds_read_b128 v[152:155], %6 offset:6144\n\t"
                    "ds_read_b128 v[156:159], %7 offset:6144"
                    : "={v[160:167]}"(wf0), "={v[128:135]}"(xf0), "={v[136:143]}"(xf1), "={v[144:151]}"(xf2), "={v[152:159]}"(xf3)
                    : "v"(lb + wa[0]), "v"(xa0), "v"(xa1));
                asm volatile(
                    "ds_read_b64_tr_b8 v[168:169], %1 offset:0\n\t"
                    "ds_read_b64_tr_b8 v[170:171], %1 offset:1024\n\t"
                    "ds_read_b64_tr_b8 v[172:173], %1 offset:2048\n\t"
                    "ds_read_b64_tr_b8 v[174:175], %1 offset:3072"
                    : "={v[168:175]}"(wf1)
                    : "v"(lb + wa[1]));
#define MFMA4(NI, WF) do { \
                    acc[NI][0] = __builtin_amdgcn_mfma_scale_f32_16x16x128_f8f6f4(WF, xf0, acc[NI][0], 1, 0, 0, 0, 0, 0); \
                    acc[NI][1] = __builtin_amdgcn_mfma_scale_f32_16x16x128_f8f6f4(WF, xf1, acc[NI][1], 1, 0, 0, 0, 0, 0); \
                    acc[NI][2] = __builtin_amdgcn_mfma_scale_f32_16x16x128_f8f6f4(WF, xf2, acc[NI][2], 1, 0, 0, 0, 0, 0); \
                    acc[NI][3] = __builtin_amdgcn_mfma_scale_f32_16x16x128_f8f6f4(WF, xf3, acc[NI][3], 1, 0, 0, 0, 0, 0); \
                    __builtin_amdgcn_sched_barrier(0); } while (0)
                asm volatile("s_waitcnt lgkmcnt(4)"
                    : "+{v[160:167]}"(wf0), "+{v[128:135]}"(xf0), "+{v[136:143]}"(xf1), "+{v[144:151]}"(xf2), "+{v[152:159]}"(xf3));
                MFMA4(0, wf0);
                asm volatile("s_waitcnt lgkmcnt(0)" : "+{v[168:175]}"(wf1));
                MFMA4(1, wf1);
                asm volatile("" ::: "memory");
                buf = (buf + 1 == NSTAGE) ? 0 : buf + 1;
            }

            STAMP(2 + 4 * round);
            const bool bad = __hip_atomic_load(tmo, __ATOMIC_RELAXED, __HIP_MEMORY_SCOPE_AGENT) == call;
            const float qnan = __builtin_bit_cast(float, 0x7fc00000u);
            if (qq == rank) {
                const __amdgpu_buffer_rsrc_t ro = make_rsrc(p.out, (unsigned)((size_t)MS * NG * 4));
                const unsigned dst0 = ((unsigned)(j * BM + wm * 64 + t16) * NG + rank * NS + ct * BN + wn * 32 + g * 4) * 4;
#pragma unroll
                for (int ni = 0; ni < 2; ++ni)
#pragma unroll
                    for (int mi = 0; mi < 4; ++mi) {
                        f32x4 y;
#pragma unroll
                        for (int e = 0; e < 4; ++e) { const float v = acc[ni][mi][e] * scale; y[e] = bad ? qnan : (v < 0.f ? 0.f : v); }
                        __builtin_amdgcn_raw_buffer_store_b128(__builtin_bit_cast(u32x4, y), ro, dst0 + (mi * 16 * NG + ni * 16) * 4, 0, 0);
                    }
            } else {
                const __amdgpu_buffer_rsrc_t rs = make_rsrc(wsme, (unsigned)WS_USED);
                const unsigned dst0 = (unsigned)(OFF_O + (size_t)qq * O_SLAB) + ((j * NCT + ct) * 8 + wave) * 8192 + lane * 16;
#pragma unroll
                for (int ni = 0; ni < 2; ++ni)
#pragma unroll
                    for (int mi = 0; mi < 4; ++mi) {
                        f32x4 y;
#pragma unroll
                        for (int e = 0; e < 4; ++e) { const float v = acc[ni][mi][e] * scale; y[e] = bad ? qnan : (v < 0.f ? 0.f : v); }
                        __builtin_amdgcn_raw_buffer_store_b128(__builtin_bit_cast(u32x4, y), rs, dst0 + (ni * 4 + mi) * 1024, 0, AUX_SYS);
                    }
                WAIT_VM0();
                if (lane == 0)
                    st_sys((unsigned*)(p.ws[qq] + OFF_FO) + ((rank * 4 + j) * NCT + ct) * 8 + wave, call);
            }
            STAMP(3 + 4 * round);
            __syncthreads();
        }
    }
}

}

extern "C" void kernel_launch(int rank, int nranks, ncclComm_t comm,
                              hipStream_t stream,
                              void* const* d_in, const int* in_sizes, int n_in,
                              void* d_out, int out_size,
                              const int* in_dtypes, int out_dtype,
                              void* const* d_out_all,
                              void* const* d_ws_all, size_t ws_bytes) {
    (void)comm; (void)in_dtypes; (void)out_dtype; (void)d_out_all;
    if (nranks != NR || n_in < 4 || rank < 0 || rank >= NR || ws_bytes < WS_USED ||
        in_sizes[0] != MG * KS || in_sizes[1] != KS * NG || out_size != MS * NG) {
        fprintf(stderr, "kernel_launch: unexpected problem shape\n");
        return;
    }
    Params p;
    p.x = (const unsigned char*)d_in[0];
    p.w = (const unsigned char*)d_in[1];
    p.sx = (const float*)d_in[2];
    p.sw = (const float*)d_in[3];
    p.out = (float*)d_out;
    for (int i = 0; i < NR; ++i) p.ws[i] = (unsigned char*)d_ws_all[i];
    p.rank = rank;
    hipLaunchKernelGGL(gemm_rs_fused, dim3(NWG), dim3(512), 0, stream, p);
    HIP_CHECK(hipGetLastError());
}
```

```cpp
#include <hip/hip_runtime.h>
#include <rccl/rccl.h>
#include <cstdio>
#include <cstdint>
#include <cstdlib>

#define HIP_CHECK(expr) do { hipError_t _e = (expr); if (_e != hipSuccess) { \
    fprintf(stderr, "HIP error %s at %s:%d\n", hipGetErrorString(_e), __FILE__, __LINE__); \
    return; } } while (0)

namespace {

constexpr int NR = 8;
constexpr int MG = 4096, KG = 4096, NG = 16384;
constexpr int KS = KG / NR;
constexpr int MS = MG / NR;
constexpr int NS = NG / NR;
constexpr int BM = 128, BN = 128, BK = 128;
constexpr int NCHUNK = MS / BM;
constexpr int NCT = NS / BN;
constexpr int NCOMM = 56, NWG = 256;
#define GPX 25
#define PULL_DEPTH 2
#define PULL_DEPTH_M1 1
#define XPULL_DEPTH 1
#define OUT_AUX 16
#define NSTAGE 5
constexpr int TILES_PER_GROUP = NCHUNK * NR * 2;

constexpr unsigned U_WP = 64;
constexpr unsigned U_W = 256;
constexpr unsigned U_XP = 16;
constexpr unsigned U_XEND = U_W + NCHUNK * NR * U_XP;
constexpr unsigned U_PULL = NCHUNK * NCT * 8;
constexpr unsigned U_ALL = U_XEND + U_PULL;
constexpr int WPIECES = 12;
constexpr unsigned LINK_WAVES = 64;

constexpr size_t MiB = 1ull << 20;
constexpr size_t OFF_FW  = 0;
constexpr size_t OFF_FWL = 128;
constexpr size_t OFF_FD  = 256;
constexpr size_t OFF_FX  = 1536;
constexpr size_t OFF_TK  = 3072;
constexpr size_t OFF_FS  = 4096;
constexpr size_t OFF_TMO = 24576;
constexpr size_t OFF_CNT = 32768;
constexpr size_t OFF_PC  = 65536;
constexpr size_t OFF_FO  = 131072;
constexpr size_t X_SLAB  = 2 * MiB;
constexpr size_t OFF_XS  = 1 * MiB;
constexpr size_t OFF_X   = OFF_XS + 2 * X_SLAB;
constexpr size_t OFF_W   = OFF_X + NR * X_SLAB;
constexpr size_t W_SLAB  = 1 * MiB;
constexpr unsigned BLK_BYTES = 256 + 8192;
constexpr size_t OFF_WS  = OFF_W + 2 * NR * W_SLAB;
constexpr size_t OFF_O   = OFF_WS + 2 * NR * W_SLAB;
constexpr size_t O_SLAB  = 5 * MiB;
constexpr size_t WS_USED = OFF_O + NR * O_SLAB;
static_assert(512 * BLK_BYTES <= O_SLAB, "slab");

constexpr unsigned long long DEADLINE_TICKS = 150000000ull;

constexpr int STAGE_BYTES = BM * BK + BK * BN;
constexpr int LDS_BYTES = NSTAGE * STAGE_BYTES;

typedef unsigned u32x4 __attribute__((ext_vector_type(4)));
typedef float f32x4 __attribute__((ext_vector_type(4)));
typedef int i32x8 __attribute__((ext_vector_type(8)));
#define LDS_AS __attribute__((address_space(3)))

struct Params {
    const unsigned char* x;
    const unsigned char* w;
    const float* sx;
    const float* sw;
    float* out;
    unsigned char* ws[NR];
    int rank;
};

constexpr int AUX_AGENT = 16;
constexpr int AUX_SYS = 17;

__device__ __forceinline__ __amdgpu_buffer_rsrc_t make_rsrc(const void* base, unsigned bytes) {
    return __builtin_amdgcn_make_buffer_rsrc(const_cast<void*>(base), 0, bytes, 0x00020000);
}
__device__ __forceinline__ unsigned ld_sys(const unsigned* p) {
    return __hip_atomic_load(p, __ATOMIC_RELAXED, __HIP_MEMORY_SCOPE_SYSTEM);
}
__device__ __forceinline__ void st_sys(unsigned* p, unsigned v) {
    __hip_atomic_store(p, v, __ATOMIC_RELAXED, __HIP_MEMORY_SCOPE_SYSTEM);
}
#define WAIT_VM0() asm volatile("s_waitcnt vmcnt(0)" ::: "memory")
#define DRAW_ASYNC(REG, ctr) do { if (lane == 0) \
    asm volatile("global_atomic_add " REG ", %0, %1, off sc0" :: "v"(ctr), "v"(1u) : REG, "memory"); } while (0)
#define STR_(x) #x
#define STR(x) STR_(x)
#define DRAW_GET(REG, N, dst) \
    asm volatile("s_waitcnt vmcnt(" STR(N) ")\n\tv_readfirstlane_b32 %0, " REG "\n\ts_nop 3" : "=s"(dst) :: REG, "memory")
#define STAMP(i) do { } while (0)
#define STAMPV(i, v) do { } while (0)

__device__ __forceinline__ bool wait_flags(const unsigned* fp, unsigned want, unsigned call, unsigned* tmo, int sleep_long) {
    const unsigned long long t0 = __builtin_amdgcn_s_memrealtime();
    for (;;) {
        const unsigned v = fp ? ld_sys(fp) : want;
        const unsigned t = __hip_atomic_load(tmo, __ATOMIC_RELAXED, __HIP_MEMORY_SCOPE_AGENT);
        if (t == call) return false;
        if (__all((int)(v - want) >= 0)) return true;
        if (__builtin_amdgcn_s_memrealtime() - t0 > DEADLINE_TICKS) {
            __hip_atomic_store(tmo, call, __ATOMIC_RELAXED, __HIP_MEMORY_SCOPE_AGENT);
            return false;
        }
        if (sleep_long) __builtin_amdgcn_s_sleep(16); else __builtin_amdgcn_s_sleep(2);
    }
}

__device__ __forceinline__ bool wait_flag64(const unsigned long long* fp, unsigned call, unsigned* tmo, unsigned& payload) {
    const unsigned long long t0 = __builtin_amdgcn_s_memrealtime();
    for (;;) {
        const unsigned long long v = __hip_atomic_load(fp, __ATOMIC_RELAXED, __HIP_MEMORY_SCOPE_SYSTEM);
        const unsigned t = __hip_atomic_load(tmo, __ATOMIC_RELAXED, __HIP_MEMORY_SCOPE_AGENT);
        const unsigned hi = __builtin_amdgcn_readfirstlane((unsigned)(v >> 32));
        payload = __builtin_amdgcn_readfirstlane((unsigned)v);
        if (t == call) return false;
        if ((int)(hi - call) >= 0) return true;
        if (__builtin_amdgcn_s_memrealtime() - t0 > DEADLINE_TICKS) {
            __hip_atomic_store(tmo, call, __ATOMIC_RELAXED, __HIP_MEMORY_SCOPE_AGENT);
            return false;
        }
        __builtin_amdgcn_s_sleep(2);
    }
}
__device__ __forceinline__ unsigned lanes_below(unsigned long long m) {
    return __builtin_amdgcn_mbcnt_hi((unsigned)(m >> 32), __builtin_amdgcn_mbcnt_lo((unsigned)m, 0u));
}
__device__ __forceinline__ int sigma_x(int r) { return ((r >> 1) & 1) | (((r >> 3) & 1) * 6); }

__global__ __launch_bounds__(512, 2) void gemm_rs_fused(Params p) {
    __shared__ __attribute__((aligned(16))) unsigned char smem[LDS_BYTES];
    const int tid = threadIdx.x;
    const int lane = tid & 63;
    const int wave = __builtin_amdgcn_readfirstlane(tid >> 6);
    const int rank = p.rank;
    unsigned char* const wsme = p.ws[rank];
    unsigned* const tmo = (unsigned*)(wsme + OFF_TMO);
    unsigned* const cntp = (unsigned*)(wsme + OFF_CNT + (size_t)blockIdx.x * 64);

    const unsigned cnt_raw = __hip_atomic_load(cntp, __ATOMIC_RELAXED, __HIP_MEMORY_SCOPE_AGENT);

    if (blockIdx.x < NCOMM) {
        const int li = blockIdx.x >> 3;
        const unsigned q = (unsigned)(rank + 1 + li) & 7u;
        unsigned char* const wsq = p.ws[q];
        unsigned* const tkp = (unsigned*)(wsme + OFF_TK + li * 128);
        const __amdgpu_buffer_rsrc_t rw = make_rsrc(p.w, (unsigned)((size_t)KS * NG));
        const __amdgpu_buffer_rsrc_t rq = make_rsrc(wsq, (unsigned)WS_USED);
        const __amdgpu_buffer_rsrc_t rm = make_rsrc(wsme, (unsigned)WS_USED);
        const __amdgpu_buffer_rsrc_t ro = make_rsrc(p.out, (unsigned)((size_t)MS * NG * 4));

        unsigned u = (blockIdx.x & 7) * 8 + wave;
        DRAW_ASYNC("v250", tkp);
        const unsigned l16 = lane * 16;
        u32x4 v[4];
#pragma unroll
        for (int i = 0; i < 4; ++i)
            v[i] = __builtin_amdgcn_raw_buffer_load_b128(rw, l16 + (i & 1) * 1024, (2 * u + (i >> 1)) * NG + q * NS, 0);
        const unsigned call = __builtin_amdgcn_readfirstlane(cnt_raw) + 1u;
        const unsigned par = call & 1u;
        const unsigned tk_base = (call - 1u) * U_ALL - LINK_WAVES;
        __syncthreads();
        if (tid == 0) __hip_atomic_store(cntp, call, __ATOMIC_RELAXED, __HIP_MEMORY_SCOPE_AGENT);
        STAMP(0);
        wait_flags(lane == 0 ? (const unsigned*)(wsme + OFF_FD) + q : nullptr, call - 1u, call, tmo, 0);

        unsigned* const pcb = (unsigned*)(wsme + OFF_PC + li * 4096);
        auto publish = [&](unsigned pc, unsigned old) {
            if (lane == 0) {
                if (pc == 34u) {
                    if (old + 1u == call * U_WP) st_sys((unsigned*)(wsq + OFF_FW) + rank, call);
                } else if (pc == 0u) {
                    if (old + 1u == call * (U_W - U_WP))
                        __hip_atomic_store((unsigned*)(wsme + OFF_FWL) + q, call, __ATOMIC_RELAXED, __HIP_MEMORY_SCOPE_AGENT);
                } else if (old + 1u == call * U_XP) {
                    __hip_atomic_store((unsigned*)(wsme + OFF_FX) + (pc - 1u) * 8 + q, call, __ATOMIC_RELAXED, __HIP_MEMORY_SCOPE_AGENT);
                }
            }
        };
#pragma unroll
        for (int i = 0; i < 4; ++i)
            __builtin_amdgcn_raw_buffer_store_b128(v[i], rq, l16 + i * 1024, (unsigned)(OFF_W + (par * NR + rank) * W_SLAB) + u * 4096, AUX_SYS);
        unsigned pc_prev = 34u;
        {
            unsigned tn;
            DRAW_GET("v250", 4, tn);
            u = tn - tk_base;
            if (u < U_ALL) DRAW_ASYNC("v250", tkp);
        }
        STAMP(1);

        if (u < U_XEND)
            wait_flags(lane < 32 + WPIECES ? (const unsigned*)(wsme + OFF_FS) + q * 64 + lane : nullptr, call, call, tmo, 0);
        while (u < U_XEND) {
            unsigned src0, dst0, pc;
            if (u < U_W) {
                pc = 0;
                src0 = (unsigned)(OFF_WS + (par * NR + rank) * W_SLAB) + u * 4096;
                dst0 = (unsigned)(OFF_W + (par * NR + q) * W_SLAB) + u * 4096;
            } else {
                const unsigned t = u - U_W, j = t >> 7, qq = (rank + 1 + (t >> 4)) & 7;
                const unsigned off = (qq * MS + j * BM) * KS + (t & 15) * 4096;
                pc = 1 + j * 8 + qq;
                src0 = (unsigned)(OFF_XS + par * X_SLAB) + off;
                dst0 = (unsigned)(OFF_X + q * X_SLAB) + off;
            }
            unsigned un = 0, old;
#pragma unroll
            for (int h = 0; h < 4; h += XPULL_DEPTH) {
#pragma unroll
                for (int f = h; f < h + XPULL_DEPTH; ++f)
                    v[f] = __builtin_amdgcn_raw_buffer_load_b128(rq, l16 + f * 1024, src0, AUX_SYS);
                if (h == 0) {
                    DRAW_GET("v250", XPULL_DEPTH, un);
                    un -= tk_base;
                    if (un < U_ALL) DRAW_ASYNC("v250", tkp);
                    DRAW_ASYNC("v251", pcb + pc_prev * 16);
                }
#pragma unroll
                for (int f = h; f < h + XPULL_DEPTH; ++f)
                    __builtin_amdgcn_raw_buffer_store_b128(v[f], rm, l16 + f * 1024, dst0, AUX_AGENT);
                if (h == 0) {
                    DRAW_GET("v251", XPULL_DEPTH, old);
                    publish(pc_prev, old);
                }
            }
            pc_prev = pc;
            u = un;
        }
        {
            unsigned old;
            WAIT_VM0();
            DRAW_ASYNC("v251", pcb + pc_prev * 16);
            DRAW_GET("v251", 0, old);
            publish(pc_prev, old);
        }
        STAMP(6);

        const unsigned olane = ((lane & 15) * NG + (lane >> 4) * 4) * 4;
        const unsigned long long* const fo = (const unsigned long long*)(wsme + OFF_FO) + q * 512;
        unsigned char* const wl = smem + wave * BLK_BYTES;
        const unsigned lds_wl = (unsigned)(uintptr_t)(LDS_AS unsigned char*)smem + wave * BLK_BYTES;
#define CLOB_Y "v96","v97","v98","v99","v100","v101","v102","v103","v104","v105","v106","v107","v108","v109","v110","v111", \
               "v112","v113","v114","v115","v116","v117","v118","v119","v120","v121","v122","v123","v124","v125","v126","v127","v128"
        bool ready = false;
        unsigned cnt = 0;
        while (u < U_ALL) {
            const unsigned blk = u - U_XEND;
            const unsigned j = blk >> 7, ct = (blk >> 3) & 15, w = blk & 7;
            bool ok = true;
            if (!ready) {
                ok = wait_flag64(fo + blk, call, tmo, cnt);
                asm volatile("" ::: "memory");
            }
            const unsigned dst0 = ((j * BM + (w >> 2) * 64) * NG + q * NS + ct * BN + (w & 3) * 32) * 4;
            unsigned un = 0;
            if (ok) {
                if (cnt > 2048u) cnt = 2048u;
                const unsigned bytes = 256u + cnt * 4u;
                const unsigned char* src = wsq + OFF_O + (size_t)rank * O_SLAB + (size_t)blk * BLK_BYTES + l16;
                for (unsigned c = 0; c * 1024u < bytes; ++c) {
                    if (c * 1024u + l16 < bytes)
                        __builtin_amdgcn_global_load_lds((const unsigned*)(src + c * 1024u), (LDS_AS unsigned*)(wl + c * 1024u), 16, 0, AUX_SYS);
                    if (c == 0) {
                        DRAW_GET("v250", 1, un);
                        un -= tk_base;
                        if (un < U_ALL) {
                            DRAW_ASYNC("v250", tkp);
                            asm volatile("global_load_dwordx2 v[252:253], %0, off sc0 sc1" :: "v"(fo + (un - U_XEND)) : "v252", "v253", "memory");
                        }
                    }
                    if (PULL_DEPTH == 1) WAIT_VM0();
                    else if (c >= PULL_DEPTH - 1) asm volatile("s_waitcnt vmcnt(" STR(PULL_DEPTH_M1) ")" ::: "memory");
                }
                WAIT_VM0();
                {
                    unsigned word = *(const unsigned*)(wl + lane * 4);
                    unsigned posb = lds_wl + 256;
                    asm volatile(
                        "v_mov_b32 v96, 0\n\tv_mov_b32 v97, 0\n\tv_mov_b32 v98, 0\n\tv_mov_b32 v99, 0\n\t"
                        "v_mov_b32 v100, 0\n\tv_mov_b32 v101, 0\n\tv_mov_b32 v102, 0\n\tv_mov_b32 v103, 0\n\t"
                        "v_mov_b32 v104, 0\n\tv_mov_b32 v105, 0\n\tv_mov_b32 v106, 0\n\tv_mov_b32 v107, 0\n\t"
                        "v_mov_b32 v108, 0\n\tv_mov_b32 v109, 0\n\tv_mov_b32 v110, 0\n\tv_mov_b32 v111, 0\n\t"
                        "v_mov_b32 v112, 0\n\tv_mov_b32 v113, 0\n\tv_mov_b32 v114, 0\n\tv_mov_b32 v115, 0\n\t"
                        "v_mov_b32 v116, 0\n\tv_mov_b32 v117, 0\n\tv_mov_b32 v118, 0\n\tv_mov_b32 v119, 0\n\t"
                        "v_mov_b32 v120, 0\n\tv_mov_b32 v121, 0\n\tv_mov_b32 v122, 0\n\tv_mov_b32 v123, 0\n\t"
                        "v_mov_b32 v124, 0\n\tv_mov_b32 v125, 0\n\tv_mov_b32 v126, 0\n\tv_mov_b32 v127, 0"
                        ::: CLOB_Y);
#define UNP(R) asm volatile( \
                        "v_cmp_gt_i32 vcc, 0, %[w]\n\t" \
                        "v_add_u32 %[w], %[w], %[w]\n\t" \
                        "s_nop 0\n\t" \
                        "v_mbcnt_lo_u32_b32 v128, vcc_lo, 0\n\t" \
                        "v_mbcnt_hi_u32_b32 v128, vcc_hi, v128\n\t" \
                        "v_lshl_add_u32 v128, v128, 2, %[p]\n\t" \
                        "v_min_u32 v128, v128, %[lim]\n\t" \
                        "s_mov_b64 exec, vcc\n\t" \
                        "ds_read_b32 v" #R ", v128\n\t" \
                        "s_mov_b64 exec, -1\n\t" \
                        "s_bcnt1_i32_b64 %[c], vcc\n\t" \
                        "s_lshl2_add_u32 %[p], %[c], %[p]\n\t" \
                        "s_nop 0" \
                        : [c] "=&s"(cc), [w] "+v"(word), [p] "+s"(posb) : [lim] "v"(lim) : "vcc", "scc", "memory", CLOB_Y)
                    unsigned cc;
                    const unsigned lim = lds_wl + BLK_BYTES - 4;
                    UNP(96);  UNP(97);  UNP(98);  UNP(99);  UNP(100); UNP(101); UNP(102); UNP(103);
                    UNP(104); UNP(105); UNP(106); UNP(107); UNP(108); UNP(109); UNP(110); UNP(111);
                    UNP(112); UNP(113); UNP(114); UNP(115); UNP(116); UNP(117); UNP(118); UNP(119);
                    UNP(120); UNP(121); UNP(122); UNP(123); UNP(124); UNP(125); UNP(126); UNP(127);
                    unsigned char* const ob = (unsigned char*)p.out + dst0 + olane;
#define OST(R0, R1, F) asm volatile("global_store_dwordx4 %0, v[" #R0 ":" #R1 "], off sc1\n\ts_nop 1" \
                        :: "v"(ob + ((F) & 3) * (16 * NG * 4) + ((F) >> 2) * 64) : "memory", CLOB_Y)
                    asm volatile("s_waitcnt lgkmcnt(0)" ::: "memory", CLOB_Y);
                    OST(96, 99, 0);   OST(100, 103, 1); OST(104, 107, 2); OST(108, 111, 3);
                    OST(112, 115, 4); OST(116, 119, 5); OST(120, 123, 6); OST(124, 127, 7);
                }
                ready = false;
                if (un < U_ALL) {
                    unsigned fhi;
                    asm volatile("s_waitcnt vmcnt(8)\n\tv_readfirstlane_b32 %0, v252\n\tv_readfirstlane_b32 %1, v253\n\ts_nop 3"
                                 : "=s"(cnt), "=s"(fhi) :: "v252", "v253", "memory");
                    ready = (int)(fhi - call) >= 0;
                }
            } else {
#pragma unroll
                for (int f = 0; f < 8; ++f)
                    __builtin_amdgcn_raw_buffer_store_b128(u32x4{0x7fc00000u, 0x7fc00000u, 0x7fc00000u, 0x7fc00000u}, ro,
                                                           olane + (f >> 2) * 64, dst0 + (f & 3) * 16 * NG * 4, OUT_AUX);
                DRAW_GET("v250", 0, un);
                un -= tk_base;
                if (un < U_ALL) DRAW_ASYNC("v250", tkp);
                ready = false;
            }
            u = un;
        }
        {
            unsigned old;
            WAIT_VM0();
            DRAW_ASYNC("v251", pcb + 33 * 16);
            DRAW_GET("v251", 0, old);
            if (lane == 0 && old + 1u == call * LINK_WAVES) st_sys((unsigned*)(wsq + OFF_FD) + rank, call);
        }
        STAMP(14);
    } else {
        unsigned call = __builtin_amdgcn_readfirstlane(cnt_raw) + 1u;
        const unsigned par = call & 1u;
        __syncthreads();
        if (tid == 0) __hip_atomic_store(cntp, call, __ATOMIC_RELAXED, __HIP_MEMORY_SCOPE_AGENT);
        STAMP(0);
        const int gi = blockIdx.x - NCOMM;
        const int grp = gi & 7;
        if (gi < NCHUNK * NR + (NR - 1) * WPIECES) {
            const __amdgpu_buffer_rsrc_t rm = make_rsrc(wsme, (unsigned)WS_USED);
            u32x4 v[8];
            if (gi < NCHUNK * NR) {
                const unsigned off = ((gi & 7) * MS + (gi >> 3) * BM) * KS + tid * 16;
                const __amdgpu_buffer_rsrc_t rx = make_rsrc(p.x, (unsigned)((size_t)MG * KS));
#pragma unroll
                for (int i = 0; i < 8; ++i) v[i] = __builtin_amdgcn_raw_buffer_load_b128(rx, off + i * 8192, 0, 0);
#pragma unroll
                for (int i = 0; i < 8; ++i)
                    __builtin_amdgcn_raw_buffer_store_b128(v[i], rm, (unsigned)(OFF_XS + par * X_SLAB) + off + i * 8192, 0, AUX_SYS);
                WAIT_VM0();
                __syncthreads();
                if (tid < NR && tid != rank) st_sys((unsigned*)(p.ws[tid] + OFF_FS) + rank * 64 + gi, call);
            } else {
                const int d = (gi - NCHUNK * NR) / WPIECES, pi = (gi - NCHUNK * NR) % WPIECES;
                const unsigned q = (unsigned)(rank + 1 + d) & 7u;
                const unsigned row = 2 * U_WP + pi * 32 + (tid >> 7), col = (tid & 127) * 16;
                const __amdgpu_buffer_rsrc_t rw = make_rsrc(p.w, (unsigned)((size_t)KS * NG));
#pragma unroll
                for (int i = 0; i < 8; ++i) v[i] = __builtin_amdgcn_raw_buffer_load_b128(rw, (row + i * 4) * NG + q * NS + col, 0, 0);
#pragma unroll
                for (int i = 0; i < 8; ++i)
                    __builtin_amdgcn_raw_buffer_store_b128(v[i], rm, (unsigned)(OFF_WS + (par * NR + q) * W_SLAB) + (row + i * 4) * NS + col, 0, AUX_SYS);
                WAIT_VM0();
                __syncthreads();
                if (tid == 0) st_sys((unsigned*)(p.ws[q] + OFF_FS) + rank * 64 + 32 + pi, call);
            }
        }
        if ((gi >> 3) >= GPX) return;
        const int wm = wave >> 2, wn = wave & 3;
        const float scale = p.sx[0] * p.sw[0];
        const unsigned char* const wsX = wsme + OFF_X;
        const unsigned char* const wsW = wsme + OFF_W + (size_t)par * NR * W_SLAB;

        unsigned xoff[2], woffR[2], woffO[2];
#pragma unroll
        for (int i = 0; i < 2; ++i) {
            const int r = wave * 16 + i * 8 + (lane >> 3);
            xoff[i] = r * KS + (((lane & 7) ^ sigma_x(r & 15)) << 4);
            const unsigned ch = (((lane & 7) ^ (((r >> 1) & 3) | (((r >> 5) & 1) << 2))) << 4);
            woffR[i] = r * NS + ch;
            woffO[i] = r * NG + ch;
        }
        const int g = lane >> 4, t16 = lane & 15;
        unsigned xa[2], wa[2];
#pragma unroll
        for (int h = 0; h < 2; ++h) xa[h] = (wm * 64 + t16) * BK + (((2 * g + h) ^ sigma_x(t16)) << 4);
        {
            const int F = ((t16 >> 2) & 3) | ((g & 1) << 2);
#pragma unroll
            for (int ni = 0; ni < 2; ++ni)
                wa[ni] = BM * BK + (32 * g + (t16 >> 1)) * BN + (((wn * 2 + ni) ^ F) << 4) + (t16 & 1) * 8;
        }
        const unsigned lds_base = (unsigned)(uintptr_t)(LDS_AS unsigned char*)smem;
        int round = 0;

        for (int idx = gi >> 3; idx < TILES_PER_GROUP; idx += GPX, ++round) {
            const int j = idx >> 4, ct = 2 * grp + (idx & 1), qq = (rank + 1 + ((idx >> 1) & 7)) & 7;
            const int R0 = qq * MS + j * BM;

            if (wave == 0) {
                const unsigned* f = nullptr;
                const int l7 = lane & 7;
                if (l7 < 7 && lane < 24) {
                    const int src = l7 + (l7 >= rank);
                    if (lane < 8)       f = (const unsigned*)(wsme + OFF_FW) + src;
                    else if (lane < 16) f = (const unsigned*)(wsme + OFF_FWL) + src;
                    else                f = (const unsigned*)(wsme + OFF_FX) + (j * 8 + qq) * 8 + src;
                }
                wait_flags(f, call, call, tmo, 1);
                __builtin_amdgcn_fence(__ATOMIC_ACQUIRE, "");
                WAIT_VM0();
            }
            __syncthreads();
            STAMP(1 + 4 * round);

            const unsigned char *xb, *wb;
            unsigned wstep, wo0, wo1;
            auto slab = [&](int ps) {
                const bool own = (ps == rank);
                xb = (own ? p.x : wsX + (size_t)ps * X_SLAB) + (size_t)R0 * KS;
                wb = (own ? p.w + rank * NS : wsW + (size_t)ps * W_SLAB) + ct * BN;
                wstep = own ? BK * NG : BK * NS;
                wo0 = own ? woffO[0] : woffR[0];
                wo1 = own ? woffO[1] : woffR[1];
            };
            auto dma = [&](unsigned lds, unsigned voff, const unsigned char* gbase) {
                asm volatile("s_mov_b32 m0, %0\n\ts_nop 0\n\tglobal_load_lds_dwordx4 %1, %2" :: "s"(lds), "v"(voff), "s"(gbase) : "memory");
            };
            auto stage = [&](int kk, int buf) {
                const unsigned lb = lds_base + buf * STAGE_BYTES + wave * 2048;
                const unsigned char* xk = xb + kk * BK;
                const unsigned char* wk = wb + kk * wstep;
                dma(lb, xoff[0], xk);
                dma(lb + 1024, xoff[1], xk);
                dma(lb + BM * BK, wo0, wk);
                dma(lb + BM * BK + 1024, wo1, wk);
            };

#define RD_X(R0, R1, A, OFF) "ds_read_b128 v[" #R0 ":" #R1 "], " A " offset:" #OFF "\n\t"
#define RD_W(R0, R1, A, OFF) "ds_read_b64_tr_b8 v[" #R0 ":" #R1 "], " A " offset:" #OFF "\n\t"
#define MM(C0, C1, W0, W1, X0, X1) "v_mfma_f32_16x16x128_f8f6f4 v[" #C0 ":" #C1 "], v[" #W0 ":" #W1 "], v[" #X0 ":" #X1 "], v[" #C0 ":" #C1 "] cbsz:1\n\t"
#define CLOB_ALL "v96","v97","v98","v99","v100","v101","v102","v103","v104","v105","v106","v107","v108","v109","v110","v111", \
               "v112","v113","v114","v115","v116","v117","v118","v119","v120","v121","v122","v123","v124","v125","v126","v127", \
               "v128","v129","v130","v131","v132","v133","v134","v135","v136","v137","v138","v139","v140","v141","v142","v143", \
               "v144","v145","v146","v147","v148","v149","v150","v151","v152","v153","v154","v155","v156","v157","v158","v159", \
               "v160","v161","v162","v163","v164","v165","v166","v167","v168","v169","v170","v171","v172","v173","v174","v175", \
               "v176","v177","v178","v179","v180","v181","v182","v183","v184","v185","v186","v187","v188","v189","v190","v191", \
               "v192","v193","v194","v195","v196","v197","v198","v199","v200","v201","v202","v203","v204","v205","v206","v207", \
               "v208","v209","v210","v211","v212","v213","v214","v215","v216","v217","v218","v219","v220","v221","v222","v223"
#define READ_A1 RD_W(160,161,"%2",0) RD_W(162,163,"%2",1024) RD_W(164,165,"%2",2048) RD_W(166,167,"%2",3072) \
                RD_X(128,131,"%0",0) RD_X(132,135,"%1",0) RD_X(136,139,"%0",2048) RD_X(140,143,"%1",2048)
#define READ_A2 RD_X(144,147,"%0",4096) RD_X(148,151,"%1",4096) RD_X(152,155,"%0",6144) RD_X(156,159,"%1",6144) \
                RD_W(168,169,"%3",0) RD_W(170,171,"%3",1024) RD_W(172,173,"%3",2048) RD_W(174,175,"%3",3072)
#define ZERO4(R) "v_mov_b32 v" #R ", 0\n\t"
            auto lds_addr = [&](int b, unsigned& a0, unsigned& a1, unsigned& a2, unsigned& a3) {
                const unsigned lb = lds_base + b * STAGE_BYTES;
                a0 = lb + xa[0]; a1 = lb + xa[1]; a2 = lb + wa[0]; a3 = lb + wa[1];
            };
            static_assert(NSTAGE == 5, "the K loop below prefetches exactly one slab (4 steps) ahead");
            slab(0);
#pragma unroll
            for (int i = 0; i < 4; ++i) stage(i, i);
            asm volatile(
                ZERO4(96) ZERO4(97) ZERO4(98) ZERO4(99) ZERO4(100) ZERO4(101) ZERO4(102) ZERO4(103)
                ZERO4(104) ZERO4(105) ZERO4(106) ZERO4(107) ZERO4(108) ZERO4(109) ZERO4(110) ZERO4(111)
                ZERO4(112) ZERO4(113) ZERO4(114) ZERO4(115) ZERO4(116) ZERO4(117) ZERO4(118) ZERO4(119)
                ZERO4(120) ZERO4(121) ZERO4(122) ZERO4(123) ZERO4(124) ZERO4(125) ZERO4(126) ZERO4(127)
                ::: CLOB_ALL);
            asm volatile("s_waitcnt vmcnt(12)" ::: "memory");
            __builtin_amdgcn_s_barrier();
            {
                unsigned a0, a1, a2, a3;
                lds_addr(0, a0, a1, a2, a3);
                asm volatile(READ_A1 READ_A2 :: "v"(a0), "v"(a1), "v"(a2), "v"(a3) : CLOB_ALL, "memory");
            }
            int buf = 0;
#define STEP(WAITN, STG, KK, CX0,CX0e,CX1,CX1e,CX2,CX2e,CX3,CX3e,CW0,CW0e,CW1,CW1e, \
             W0,W1,W2,W3,W4,W5,W6,W7, X0,X1,X2,X3,X4,X5,X6,X7,X8,X9,XA,XB,XC,XD,XE,XF, V0,V1,V2,V3,V4,V5,V6,V7) \
            do { \
                int nb = buf + 1; if (nb == NSTAGE) nb = 0; \
                unsigned a0, a1, a2, a3; \
                lds_addr(nb, a0, a1, a2, a3); \
                asm volatile("s_waitcnt vmcnt(" #WAITN ")" ::: "memory"); \
                __builtin_amdgcn_s_barrier(); \
                asm volatile("s_waitcnt lgkmcnt(0)\n\t" \
                    MM(96,99, CW0, CW0e, CX0, CX0e) RD_W(W0,W1,"%2",0) RD_W(W2,W3,"%2",1024) \
                    MM(100,103, CW0, CW0e, CX1, CX1e) RD_W(W4,W5,"%2",2048) RD_W(W6,W7,"%2",3072) \
                    MM(104,107, CW0, CW0e, CX2, CX2e) RD_X(X0,X1,"%0",0) RD_X(X2,X3,"%1",0) \
                    MM(108,111, CW0, CW0e, CX3, CX3e) RD_X(X4,X5,"%0",2048) RD_X(X6,X7,"%1",2048) \
                    :: "v"(a0), "v"(a1), "v"(a2) : CLOB_ALL, "memory"); \
                if (STG) { int pb = buf - 1; if (pb < 0) pb += NSTAGE; stage(KK, pb); } \
                asm volatile( \
                    MM(112,115, CW1, CW1e, CX0, CX0e) RD_X(X8,X9,"%0",4096) RD_X(XA,XB,"%1",4096) \
                    MM(116,119, CW1, CW1e, CX1, CX1e) RD_X(XC,XD,"%0",6144) RD_X(XE,XF,"%1",6144) \
                    MM(120,123, CW1, CW1e, CX2, CX2e) RD_W(V0,V1,"%2",0) RD_W(V2,V3,"%2",1024) \
                    MM(124,127, CW1, CW1e, CX3, CX3e) RD_W(V4,V5,"%2",2048) RD_W(V6,V7,"%2",3072) \
                    :: "v"(a0), "v"(a1), "v"(a3) : CLOB_ALL, "memory"); \
                buf = nb; \
            } while (0)
#define STEP_A(WAITN, STG, KK) STEP(WAITN, STG, KK, 128,135,136,143,144,151,152,159,160,167,168,175, \
                     208,209,210,211,212,213,214,215, 176,179,180,183,184,187,188,191,192,195,196,199,200,203,204,207, \
                     216,217,218,219,220,221,222,223)
#define STEP_B(WAITN, STG, KK) STEP(WAITN, STG, KK, 176,183,184,191,192,199,200,207,208,215,216,223, \
                     160,161,162,163,164,165,166,167, 128,131,132,135,136,139,140,143,144,147,148,151,152,155,156,159, \
                     168,169,170,171,172,173,174,175)
            for (int ps = 0; ps < NR - 1; ++ps) {
                slab(ps + 1);
                STEP_A(8, 1, 0);
                STEP_B(8, 1, 1);
                STEP_A(8, 1, 2);
                STEP_B(8, 1, 3);
            }
            STEP_A(8, 0, 0);
            STEP_B(4, 0, 1);
            STEP_A(0, 0, 2);
            asm volatile("s_waitcnt lgkmcnt(0)\n\t"
                MM(96,99, 208,215, 176,183) MM(100,103, 208,215, 184,191) MM(104,107, 208,215, 192,199) MM(108,111, 208,215, 200,207)
                MM(112,115, 216,223, 176,183) MM(116,119, 216,223, 184,191) MM(120,123, 216,223, 192,199) MM(124,127, 216,223, 200,207)
                "s_nop 15\n\ts_nop 15"
                ::: CLOB_ALL, "memory");
            f32x4 acc[2][4];
#define GET4(T, R0, R1, R2, R3) do { float y0, y1, y2, y3; \
                asm volatile("v_mul_f32 %0, %4, v" #R0 "\n\tv_mul_f32 %1, %4, v" #R1 "\n\tv_mul_f32 %2, %4, v" #R2 "\n\tv_mul_f32 %3, %4, v" #R3 \
                    : "=&v"(y0), "=&v"(y1), "=&v"(y2), "=&v"(y3) : "v"(scale) : CLOB_ALL); \
                T = f32x4{y0, y1, y2, y3}; } while (0)
            GET4(acc[0][0], 96, 97, 98, 99);     GET4(acc[0][1], 100, 101, 102, 103);
            GET4(acc[0][2], 104, 105, 106, 107); GET4(acc[0][3], 108, 109, 110, 111);
            GET4(acc[1][0], 112, 113, 114, 115); GET4(acc[1][1], 116, 117, 118, 119);
            GET4(acc[1][2], 120, 121, 122, 123); GET4(acc[1][3], 124, 125, 126, 127);

            STAMP(2 + 4 * round);
            const bool bad = __hip_atomic_load(tmo, __ATOMIC_RELAXED, __HIP_MEMORY_SCOPE_AGENT) == call;
            if (bad) {
                const float qnan = __builtin_bit_cast(float, 0x7fc00000u);
#pragma unroll
                for (int a = 0; a < 2; ++a)
#pragma unroll
                    for (int b = 0; b < 4; ++b) acc[a][b] = f32x4{qnan, qnan, qnan, qnan};
            }
            if (qq == rank) {
                const __amdgpu_buffer_rsrc_t ro = make_rsrc(p.out, (unsigned)((size_t)MS * NG * 4));
                const unsigned dst0 = ((unsigned)(j * BM + wm * 64 + t16) * NG + rank * NS + ct * BN + wn * 32 + g * 4) * 4;
#pragma unroll
                for (int ni = 0; ni < 2; ++ni)
#pragma unroll
                    for (int mi = 0; mi < 4; ++mi) {
                        f32x4 y;
#pragma unroll
                        for (int e = 0; e < 4; ++e) { const float v = acc[ni][mi][e]; y[e] = v < 0.f ? 0.f : v; }
                        __builtin_amdgcn_raw_buffer_store_b128(__builtin_bit_cast(u32x4, y), ro, dst0 + (mi * 16 * NG + ni * 16) * 4, 0, 0);
                    }
            } else {
                __syncthreads();
                unsigned char* const wl = smem + wave * BLK_BYTES;
                const unsigned val0 = lds_base + wave * BLK_BYTES + 256;
                unsigned posb = val0, word = 0;
#pragma unroll
                for (int ni = 0; ni < 2; ++ni)
#pragma unroll
                    for (int mi = 0; mi < 4; ++mi)
#pragma unroll
                        for (int e = 0; e < 4; ++e) {
                            unsigned t, c;
                            unsigned long long co;
                            asm volatile(
                                "v_cmp_nge_f32 vcc, 0, %[v]\n\t"
                                "s_nop 1\n\t"
                                "v_addc_co_u32 %[w], %[co], %[w], %[w], vcc\n\t"
                                "v_mbcnt_lo_u32_b32 %[t], vcc_lo, 0\n\t"
                                "v_mbcnt_hi_u32_b32 %[t], vcc_hi, %[t]\n\t"
                                "v_lshl_add_u32 %[t], %[t], 2, %[p]\n\t"
                                "s_mov_b64 exec, vcc\n\t"
                                "ds_write_b32 %[t], %[v]\n\t"
                                "s_mov_b64 exec, -1\n\t"
                                "s_bcnt1_i32_b64 %[c], vcc\n\t"
                                "s_lshl2_add_u32 %[p], %[c], %[p]\n\t"
                                "s_nop 0"
                                : [t] "=&v"(t), [c] "=&s"(c), [co] "=&s"(co), [w] "+v"(word), [p] "+s"(posb)
                                : [v] "v"(acc[ni][mi][e])
                                : "vcc", "scc", "memory");
                        }
                const unsigned pos = (posb - val0) >> 2;
                *(unsigned*)(wl + lane * 4) = word;
                asm volatile("s_waitcnt lgkmcnt(0)" ::: "memory");
                const unsigned bytes = 256u + pos * 4u;
                const __amdgpu_buffer_rsrc_t rs = make_rsrc(wsme, (unsigned)WS_USED);
                const unsigned blk = (j * NCT + ct) * 8 + wave;
                const unsigned dst0 = (unsigned)(OFF_O + (size_t)qq * O_SLAB) + blk * BLK_BYTES;
#pragma unroll
                for (int c = 0; c < 9; ++c) {
                    const unsigned off = c * 1024 + lane * 16;
                    if (off < bytes)
                        __builtin_amdgcn_raw_buffer_store_b128(*(const u32x4*)(wl + off), rs, off, dst0, AUX_SYS);
                }
                WAIT_VM0();
                if (lane == 0)
                    __hip_atomic_store((unsigned long long*)(p.ws[qq] + OFF_FO) + rank * 512 + blk,
                                       ((unsigned long long)call << 32) | pos, __ATOMIC_RELAXED, __HIP_MEMORY_SCOPE_SYSTEM);
            }
            STAMP(3 + 4 * round);
            __syncthreads();
        }
    }
}

}

extern "C" void kernel_launch(int rank, int nranks, ncclComm_t comm,
                              hipStream_t stream,
                              void* const* d_in, const int* in_sizes, int n_in,
                              void* d_out, int out_size,
                              const int* in_dtypes, int out_dtype,
                              void* const* d_out_all,
                              void* const* d_ws_all, size_t ws_bytes) {
    (void)comm; (void)in_dtypes; (void)out_dtype; (void)d_out_all;
    if (nranks != NR || n_in < 4 || rank < 0 || rank >= NR || ws_bytes < WS_USED ||
        in_sizes[0] != MG * KS || in_sizes[1] != KS * NG || out_size != MS * NG) {
        fprintf(stderr, "kernel_launch: unexpected problem shape\n");
        return;
    }
    Params p;
    p.x = (const unsigned char*)d_in[0];
    p.w = (const unsigned char*)d_in[1];
    p.sx = (const float*)d_in[2];
    p.sw = (const float*)d_in[3];
    p.out = (float*)d_out;
    for (int i = 0; i < NR; ++i) p.ws[i] = (unsigned char*)d_ws_all[i];
    p.rank = rank;
    hipLaunchKernelGGL(gemm_rs_fused, dim3(NWG), dim3(512), 0, stream, p);
    HIP_CHECK(hipGetLastError());
}
```

```cpp
#include <hip/hip_runtime.h>
#include <rccl/rccl.h>
#include <cstdio>
#include <cstdint>

#define HIP_CHECK(expr) do { hipError_t _e = (expr); if (_e != hipSuccess) { \
    fprintf(stderr, "HIP error %s at %s:%d\n", hipGetErrorString(_e), __FILE__, __LINE__); \
    return; } } while (0)

namespace {

constexpr int NR = 8;
constexpr int MG = 4096, KG = 4096, NG = 16384;
constexpr int KS = KG / NR;
constexpr int MS = MG / NR;
constexpr int NS = NG / NR;
constexpr int BM = 128, BN = 128, BK = 128;
constexpr int NCHUNK = MS / BM;
constexpr int NCT = NS / BN;
constexpr int NCOMM = 56, NWG = 256;
constexpr int GPX = 25;
constexpr int NSTAGE = 5;
#define PULL_DEPTH 2
#define PULL_DEPTH_M1 1
#define OUT_AUX 16
constexpr int TILES_PER_GROUP = NCHUNK * NR * 2;

constexpr unsigned U_WP = 64;
constexpr unsigned U_W = 256;
constexpr unsigned U_XP = 16;
constexpr unsigned U_XEND = U_W + NCHUNK * NR * U_XP;
constexpr unsigned U_PULL = NCHUNK * NCT * 8;
constexpr unsigned U_ALL = U_XEND + U_PULL;
constexpr int WPIECES = (256 - U_WP) / 16;
constexpr unsigned LINK_WAVES = 64;

constexpr size_t MiB = 1ull << 20;
constexpr size_t OFF_FW  = 0;
constexpr size_t OFF_FWL = 128;
constexpr size_t OFF_FD  = 256;
constexpr size_t OFF_FX  = 1536;
constexpr size_t OFF_TK  = 3072;
constexpr size_t OFF_FS  = 4096;
constexpr size_t OFF_TMO = 24576;
constexpr size_t OFF_CNT = 32768;
constexpr size_t OFF_PC  = 65536;
constexpr size_t OFF_FO  = 131072;
constexpr size_t X_SLAB  = 2 * MiB;
constexpr size_t OFF_XS  = 1 * MiB;
constexpr size_t OFF_X   = OFF_XS + 2 * X_SLAB;
constexpr size_t OFF_W   = OFF_X + NR * X_SLAB;
constexpr size_t W_SLAB  = 1 * MiB;
constexpr unsigned BLK_BYTES = 256 + 8192;
constexpr size_t OFF_WS  = OFF_W + 2 * NR * W_SLAB;
constexpr size_t OFF_O   = OFF_WS + 2 * NR * W_SLAB;
constexpr size_t O_SLAB  = 5 * MiB;
constexpr size_t WS_USED = OFF_O + NR * O_SLAB;
static_assert(512 * BLK_BYTES <= O_SLAB, "slab");

constexpr unsigned long long DEADLINE_TICKS = 150000000ull;

constexpr int STAGE_BYTES = BM * BK + BK * BN;
constexpr int LDS_BYTES = NSTAGE * STAGE_BYTES;

typedef unsigned u32x4 __attribute__((ext_vector_type(4)));
typedef float f32x4 __attribute__((ext_vector_type(4)));
#define LDS_AS __attribute__((address_space(3)))

struct Params {
    const unsigned char* x;
    const unsigned char* w;
    const float* sx;
    const float* sw;
    float* out;
    unsigned char* ws[NR];
    int rank;
};

constexpr int AUX_AGENT = 16;
constexpr int AUX_SYS = 17;

__device__ __forceinline__ __amdgpu_buffer_rsrc_t make_rsrc(const void* base, unsigned bytes) {
    return __builtin_amdgcn_make_buffer_rsrc(const_cast<void*>(base), 0, bytes, 0x00020000);
}
__device__ __forceinline__ unsigned ld_sys(const unsigned* p) {
    return __hip_atomic_load(p, __ATOMIC_RELAXED, __HIP_MEMORY_SCOPE_SYSTEM);
}
__device__ __forceinline__ void st_sys(unsigned* p, unsigned v) {
    __hip_atomic_store(p, v, __ATOMIC_RELAXED, __HIP_MEMORY_SCOPE_SYSTEM);
}
#define WAIT_VM0() asm volatile("s_waitcnt vmcnt(0)" ::: "memory")
#define DRAW_ASYNC(REG, ctr) do { if (lane == 0) \
    asm volatile("global_atomic_add " REG ", %0, %1, off sc0" :: "v"(ctr), "v"(1u) : REG, "memory"); } while (0)
#define STR_(x) #x
#define STR(x) STR_(x)
#define DRAW_GET(REG, N, dst) \
    asm volatile("s_waitcnt vmcnt(" STR(N) ")\n\tv_readfirstlane_b32 %0, " REG "\n\ts_nop 3" : "=s"(dst) :: REG, "memory")

__device__ __forceinline__ bool wait_flags(const unsigned* fp, unsigned want, unsigned call, unsigned* tmo, int sleep_long) {
    const unsigned long long t0 = __builtin_amdgcn_s_memrealtime();
    for (;;) {
        const unsigned v = fp ? ld_sys(fp) : want;
        const unsigned t = __hip_atomic_load(tmo, __ATOMIC_RELAXED, __HIP_MEMORY_SCOPE_AGENT);
        if (t == call) return false;
        if (__all((int)(v - want) >= 0)) return true;
        if (__builtin_amdgcn_s_memrealtime() - t0 > DEADLINE_TICKS) {
            __hip_atomic_store(tmo, call, __ATOMIC_RELAXED, __HIP_MEMORY_SCOPE_AGENT);
            return false;
        }
        if (sleep_long) __builtin_amdgcn_s_sleep(16); else __builtin_amdgcn_s_sleep(2);
    }
}

__device__ __forceinline__ unsigned long long wait_staged(const unsigned* fs, unsigned need, unsigned call, unsigned* tmo, int lane) {
    const unsigned long long t0 = __builtin_amdgcn_s_memrealtime();
    for (;;) {
        const unsigned v = ld_sys(fs + lane);
        const unsigned t = __hip_atomic_load(tmo, __ATOMIC_RELAXED, __HIP_MEMORY_SCOPE_AGENT);
        const unsigned long long m = __builtin_amdgcn_ballot_w64((int)(v - call) >= 0);
        if (t == call) return ~0ull;
        if ((m >> need) & 1ull) return m;
        if (__builtin_amdgcn_s_memrealtime() - t0 > DEADLINE_TICKS) {
            __hip_atomic_store(tmo, call, __ATOMIC_RELAXED, __HIP_MEMORY_SCOPE_AGENT);
            return ~0ull;
        }
        __builtin_amdgcn_s_sleep(2);
    }
}
__device__ __forceinline__ bool wait_flag64(const unsigned long long* fp, unsigned call, unsigned* tmo, unsigned& payload) {
    const unsigned long long t0 = __builtin_amdgcn_s_memrealtime();
    for (;;) {
        const unsigned long long v = __hip_atomic_load(fp, __ATOMIC_RELAXED, __HIP_MEMORY_SCOPE_SYSTEM);
        const unsigned t = __hip_atomic_load(tmo, __ATOMIC_RELAXED, __HIP_MEMORY_SCOPE_AGENT);
        const unsigned hi = __builtin_amdgcn_readfirstlane((unsigned)(v >> 32));
        payload = __builtin_amdgcn_readfirstlane((unsigned)v);
        if (t == call) return false;
        if ((int)(hi - call) >= 0) return true;
        if (__builtin_amdgcn_s_memrealtime() - t0 > DEADLINE_TICKS) {
            __hip_atomic_store(tmo, call, __ATOMIC_RELAXED, __HIP_MEMORY_SCOPE_AGENT);
            return false;
        }
        __builtin_amdgcn_s_sleep(2);
    }
}
__device__ __forceinline__ int sigma_x(int r) { return ((r >> 1) & 1) | (((r >> 3) & 1) * 6); }

__global__ __launch_bounds__(512, 2) void gemm_rs_fused(Params p) {
    __shared__ __attribute__((aligned(16))) unsigned char smem[LDS_BYTES];
    const int tid = threadIdx.x;
    const int lane = tid & 63;
    const int wave = __builtin_amdgcn_readfirstlane(tid >> 6);
    const int rank = p.rank;
    unsigned char* const wsme = p.ws[rank];
    unsigned* const tmo = (unsigned*)(wsme + OFF_TMO);
    unsigned* const cntp = (unsigned*)(wsme + OFF_CNT + (size_t)blockIdx.x * 64);

    const unsigned cnt_raw = __hip_atomic_load(cntp, __ATOMIC_RELAXED, __HIP_MEMORY_SCOPE_AGENT);

    if (blockIdx.x < NCOMM) {
        const int li = blockIdx.x >> 3;
        const unsigned q = (unsigned)(rank + 1 + li) & 7u;
        unsigned char* const wsq = p.ws[q];
        unsigned* const tkp = (unsigned*)(wsme + OFF_TK + li * 128);
        const __amdgpu_buffer_rsrc_t rw = make_rsrc(p.w, (unsigned)((size_t)KS * NG));
        const __amdgpu_buffer_rsrc_t rq = make_rsrc(wsq, (unsigned)WS_USED);
        const __amdgpu_buffer_rsrc_t rm = make_rsrc(wsme, (unsigned)WS_USED);
        const __amdgpu_buffer_rsrc_t ro = make_rsrc(p.out, (unsigned)((size_t)MS * NG * 4));

        unsigned u = (blockIdx.x & 7) * 8 + wave;
        DRAW_ASYNC("v250", tkp);
        const unsigned fd_early = ld_sys((const unsigned*)(wsme + OFF_FD) + q);
        const unsigned l16 = lane * 16;
        u32x4 v[4];
#pragma unroll
        for (int i = 0; i < 4; ++i)
            v[i] = __builtin_amdgcn_raw_buffer_load_b128(rw, l16 + (i & 1) * 1024, (2 * u + (i >> 1)) * NG + q * NS, 0);
        const unsigned call = __builtin_amdgcn_readfirstlane(cnt_raw) + 1u;
        const unsigned par = call & 1u;
        const unsigned tk_base = (call - 1u) * (U_ALL - U_WP + LINK_WAVES) - U_WP;
        __syncthreads();
        if (tid == 0) __hip_atomic_store(cntp, call, __ATOMIC_RELAXED, __HIP_MEMORY_SCOPE_AGENT);
        if ((int)(__builtin_amdgcn_readfirstlane(fd_early) - (call - 1u)) < 0)
            wait_flags(lane == 0 ? (const unsigned*)(wsme + OFF_FD) + q : nullptr, call - 1u, call, tmo, 0);

        unsigned* const pcb = (unsigned*)(wsme + OFF_PC + li * 4096);
        auto publish = [&](unsigned pc, unsigned old) {
            if (lane == 0) {
                if (pc == 34u) {
                    if (old + 1u == call * LINK_WAVES) st_sys((unsigned*)(wsq + OFF_FW) + rank, call);
                } else if (pc == 0u) {
                    if (old + 1u == call * (U_W - U_WP))
                        __hip_atomic_store((unsigned*)(wsme + OFF_FWL) + q, call, __ATOMIC_RELAXED, __HIP_MEMORY_SCOPE_AGENT);
                } else if (old + 1u == call * U_XP) {
                    __hip_atomic_store((unsigned*)(wsme + OFF_FX) + (pc - 1u) * 8 + q, call, __ATOMIC_RELAXED, __HIP_MEMORY_SCOPE_AGENT);
                }
            }
        };
#pragma unroll
        for (int i = 0; i < 4; ++i)
            __builtin_amdgcn_raw_buffer_store_b128(v[i], rq, l16 + i * 1024, (unsigned)(OFF_W + (par * NR + rank) * W_SLAB) + u * 4096, AUX_SYS);
        unsigned pc_prev = 34u;
        {
            unsigned tn;
            DRAW_GET("v250", 4, tn);
            u = tn - tk_base;
            if (u < U_ALL) DRAW_ASYNC("v250", tkp);
        }

        unsigned long long staged = 0;
        const unsigned long long* const fo = (const unsigned long long*)(wsme + OFF_FO) + q * 512;
        bool pre = false;
        while (u < U_XEND) {
            unsigned src0, dst0, pc, need;
            if (u < U_W) {
                pc = 0;
                need = 32 + ((u - U_WP) >> 4);
                src0 = (unsigned)(OFF_WS + (par * NR + rank) * W_SLAB) + u * 4096;
                dst0 = (unsigned)(OFF_W + (par * NR + q) * W_SLAB) + u * 4096;
            } else {
                const unsigned t = u - U_W, j = t >> 7, qq = (rank + 1 + (t >> 4)) & 7;
                const unsigned off = (qq * MS + j * BM) * KS + (t & 15) * 4096;
                pc = 1 + j * 8 + qq;
                need = j * 8 + qq;
                src0 = (unsigned)(OFF_XS + par * X_SLAB) + off;
                dst0 = (unsigned)(OFF_X + q * X_SLAB) + off;
            }
            if (!((staged >> need) & 1ull)) staged = wait_staged((const unsigned*)(wsme + OFF_FS) + q * 64, need, call, tmo, lane);
            unsigned un = 0, old;
            v[0] = __builtin_amdgcn_raw_buffer_load_b128(rq, l16, src0, AUX_SYS);
            DRAW_GET("v250", 1, un);
            un -= tk_base;
            if (un < U_ALL) DRAW_ASYNC("v250", tkp);
            DRAW_ASYNC("v251", pcb + pc_prev * 16);
            if (un >= U_XEND && un < U_ALL)
                asm volatile("global_load_dwordx2 v[252:253], %0, off sc0 sc1" :: "v"(fo + (un - U_XEND)) : "v252", "v253", "memory");
            __builtin_amdgcn_raw_buffer_store_b128(v[0], rm, l16, dst0, AUX_AGENT);
            DRAW_GET("v251", 1, old);
            publish(pc_prev, old);
#pragma unroll
            for (int f = 1; f < 4; ++f) {
                v[f] = __builtin_amdgcn_raw_buffer_load_b128(rq, l16 + f * 1024, src0, AUX_SYS);
                __builtin_amdgcn_raw_buffer_store_b128(v[f], rm, l16 + f * 1024, dst0, AUX_AGENT);
            }
            pre = un >= U_XEND && un < U_ALL;
            pc_prev = pc;
            u = un;
        }
        {
            unsigned old;
            WAIT_VM0();
            DRAW_ASYNC("v251", pcb + pc_prev * 16);
            DRAW_GET("v251", 0, old);
            publish(pc_prev, old);
        }

        const unsigned olane = ((lane & 15) * NG + (lane >> 4) * 4) * 4;
        unsigned char* const wl = smem + wave * BLK_BYTES;
        const unsigned lds_wl = (unsigned)(uintptr_t)(LDS_AS unsigned char*)smem + wave * BLK_BYTES;
#define CLOB_Y "v96","v97","v98","v99","v100","v101","v102","v103","v104","v105","v106","v107","v108","v109","v110","v111", \
               "v112","v113","v114","v115","v116","v117","v118","v119","v120","v121","v122","v123","v124","v125","v126","v127","v128"
        bool ready = false;
        unsigned cnt = 0;
        if (pre) {
            unsigned fhi;
            asm volatile("v_readfirstlane_b32 %0, v252\n\tv_readfirstlane_b32 %1, v253\n\ts_nop 3"
                         : "=s"(cnt), "=s"(fhi) :: "v252", "v253", "memory");
            ready = (int)(fhi - call) >= 0;
        }
        while (u < U_ALL) {
            const unsigned blk = u - U_XEND;
            const unsigned j = blk >> 7, ct = (blk >> 3) & 15, w = blk & 7;
            bool ok = true;
            if (!ready) {
                ok = wait_flag64(fo + blk, call, tmo, cnt);
                asm volatile("" ::: "memory");
            }
            const unsigned dst0 = ((j * BM + (w >> 2) * 64) * NG + q * NS + ct * BN + (w & 3) * 32) * 4;
            unsigned un = 0;
            if (ok) {
                if (cnt > 2048u) cnt = 2048u;
                const unsigned bytes = 256u + cnt * 4u;
                const unsigned char* src = wsq + OFF_O + (size_t)rank * O_SLAB + (size_t)blk * BLK_BYTES + l16;
                for (unsigned c = 0; c * 1024u < bytes; ++c) {
                    if (c * 1024u + l16 < bytes)
                        __builtin_amdgcn_global_load_lds((const unsigned*)(src + c * 1024u), (LDS_AS unsigned*)(wl + c * 1024u), 16, 0, AUX_SYS);
                    if (c == 0) {
                        DRAW_GET("v250", 1, un);
                        un -= tk_base;
                        if (un < U_ALL) {
                            DRAW_ASYNC("v250", tkp);
                            asm volatile("global_load_dwordx2 v[252:253], %0, off sc0 sc1" :: "v"(fo + (un - U_XEND)) : "v252", "v253", "memory");
                        }
                    }
                    if (PULL_DEPTH == 1) WAIT_VM0();
                    else if (c >= PULL_DEPTH - 1) asm volatile("s_waitcnt vmcnt(" STR(PULL_DEPTH_M1) ")" ::: "memory");
                }
                WAIT_VM0();
                {
                    unsigned word = *(const unsigned*)(wl + lane * 4);
                    unsigned posb = lds_wl + 256;
                    asm volatile(
                        "v_mov_b32 v96, 0\n\tv_mov_b32 v97, 0\n\tv_mov_b32 v98, 0\n\tv_mov_b32 v99, 0\n\t"
                        "v_mov_b32 v100, 0\n\tv_mov_b32 v101, 0\n\tv_mov_b32 v102, 0\n\tv_mov_b32 v103, 0\n\t"
                        "v_mov_b32 v104, 0\n\tv_mov_b32 v105, 0\n\tv_mov_b32 v106, 0\n\tv_mov_b32 v107, 0\n\t"
                        "v_mov_b32 v108, 0\n\tv_mov_b32 v109, 0\n\tv_mov_b32 v110, 0\n\tv_mov_b32 v111, 0\n\t"
                        "v_mov_b32 v112, 0\n\tv_mov_b32 v113, 0\n\tv_mov_b32 v114, 0\n\tv_mov_b32 v115, 0\n\t"
                        "v_mov_b32 v116, 0\n\tv_mov_b32 v117, 0\n\tv_mov_b32 v118, 0\n\tv_mov_b32 v119, 0\n\t"
                        "v_mov_b32 v120, 0\n\tv_mov_b32 v121, 0\n\tv_mov_b32 v122, 0\n\tv_mov_b32 v123, 0\n\t"
                        "v_mov_b32 v124, 0\n\tv_mov_b32 v125, 0\n\tv_mov_b32 v126, 0\n\tv_mov_b32 v127, 0"
                        ::: CLOB_Y);
#define UNP(R) asm volatile( \
                        "v_cmp_gt_i32 vcc, 0, %[w]\n\t" \
                        "v_add_u32 %[w], %[w], %[w]\n\t" \
                        "s_nop 0\n\t" \
                        "v_mbcnt_lo_u32_b32 v128, vcc_lo, 0\n\t" \
                        "v_mbcnt_hi_u32_b32 v128, vcc_hi, v128\n\t" \
                        "v_lshl_add_u32 v128, v128, 2, %[p]\n\t" \
                        "v_min_u32 v128, v128, %[lim]\n\t" \
                        "s_mov_b64 exec, vcc\n\t" \
                        "ds_read_b32 v" #R ", v128\n\t" \
                        "s_mov_b64 exec, -1\n\t" \
                        "s_bcnt1_i32_b64 %[c], vcc\n\t" \
                        "s_lshl2_add_u32 %[p], %[c], %[p]\n\t" \
                        "s_nop 0" \
                        : [c] "=&s"(cc), [w] "+v"(word), [p] "+s"(posb) : [lim] "v"(lim) : "vcc", "scc", "memory", CLOB_Y)
                    unsigned cc;
                    const unsigned lim = lds_wl + BLK_BYTES - 4;
                    UNP(96);  UNP(97);  UNP(98);  UNP(99);  UNP(100); UNP(101); UNP(102); UNP(103);
                    UNP(104); UNP(105); UNP(106); UNP(107); UNP(108); UNP(109); UNP(110); UNP(111);
                    UNP(112); UNP(113); UNP(114); UNP(115); UNP(116); UNP(117); UNP(118); UNP(119);
                    UNP(120); UNP(121); UNP(122); UNP(123); UNP(124); UNP(125); UNP(126); UNP(127);
                    unsigned char* const ob = (unsigned char*)p.out + dst0 + olane;
#define OST(R0, R1, F) asm volatile("global_store_dwordx4 %0, v[" #R0 ":" #R1 "], off sc1\n\ts_nop 1" \
                        :: "v"(ob + ((F) & 3) * (16 * NG * 4) + ((F) >> 2) * 64) : "memory", CLOB_Y)
                    asm volatile("s_waitcnt lgkmcnt(0)" ::: "memory", CLOB_Y);
                    OST(96, 99, 0);   OST(112, 115, 4); OST(100, 103, 1); OST(116, 119, 5);
                    OST(104, 107, 2); OST(120, 123, 6); OST(108, 111, 3); OST(124, 127, 7);
                }
                ready = false;
                if (un < U_ALL) {
                    unsigned fhi;
                    asm volatile("s_waitcnt vmcnt(8)\n\tv_readfirstlane_b32 %0, v252\n\tv_readfirstlane_b32 %1, v253\n\ts_nop 3"
                                 : "=s"(cnt), "=s"(fhi) :: "v252", "v253", "memory");
                    ready = (int)(fhi - call) >= 0;
                }
            } else {
#pragma unroll
                for (int f = 0; f < 8; ++f)
                    __builtin_amdgcn_raw_buffer_store_b128(u32x4{0x7fc00000u, 0x7fc00000u, 0x7fc00000u, 0x7fc00000u}, ro,
                                                           olane + (f >> 2) * 64, dst0 + (f & 3) * 16 * NG * 4, OUT_AUX);
                DRAW_GET("v250", 0, un);
                un -= tk_base;
                if (un < U_ALL) DRAW_ASYNC("v250", tkp);
                ready = false;
            }
            u = un;
        }
        {
            unsigned old;
            WAIT_VM0();
            DRAW_ASYNC("v251", pcb + 33 * 16);
            DRAW_GET("v251", 0, old);
            if (lane == 0 && old + 1u == call * LINK_WAVES) st_sys((unsigned*)(wsq + OFF_FD) + rank, call);
        }
    } else {
        unsigned call = __builtin_amdgcn_readfirstlane(cnt_raw) + 1u;
        const unsigned par = call & 1u;
        __syncthreads();
        if (tid == 0) __hip_atomic_store(cntp, call, __ATOMIC_RELAXED, __HIP_MEMORY_SCOPE_AGENT);
        const int gi = blockIdx.x - NCOMM;
        const int grp = gi & 7;
        if (gi < NCHUNK * NR + (NR - 1) * WPIECES) {
            const __amdgpu_buffer_rsrc_t rm = make_rsrc(wsme, (unsigned)WS_USED);
            u32x4 v[8];
            if (gi >= (NR - 1) * WPIECES) {
                const int xi = gi - (NR - 1) * WPIECES;
                const unsigned off = ((xi & 7) * MS + (xi >> 3) * BM) * KS + tid * 16;
                const __amdgpu_buffer_rsrc_t rx = make_rsrc(p.x, (unsigned)((size_t)MG * KS));
#pragma unroll
                for (int i = 0; i < 8; ++i) v[i] = __builtin_amdgcn_raw_buffer_load_b128(rx, off + i * 8192, 0, 0);
#pragma unroll
                for (int i = 0; i < 8; ++i)
                    __builtin_amdgcn_raw_buffer_store_b128(v[i], rm, (unsigned)(OFF_XS + par * X_SLAB) + off + i * 8192, 0, AUX_SYS);
                WAIT_VM0();
                __syncthreads();
                if (tid < NR && tid != rank) st_sys((unsigned*)(p.ws[tid] + OFF_FS) + rank * 64 + xi, call);
            } else {
                const int pi = gi / (NR - 1), d = gi % (NR - 1);
                const unsigned q = (unsigned)(rank + 1 + d) & 7u;
                const unsigned row = 2 * U_WP + pi * 32 + (tid >> 7), col = (tid & 127) * 16;
                const __amdgpu_buffer_rsrc_t rw = make_rsrc(p.w, (unsigned)((size_t)KS * NG));
#pragma unroll
                for (int i = 0; i < 8; ++i) v[i] = __builtin_amdgcn_raw_buffer_load_b128(rw, (row + i * 4) * NG + q * NS + col, 0, 0);
#pragma unroll
                for (int i = 0; i < 8; ++i)
                    __builtin_amdgcn_raw_buffer_store_b128(v[i], rm, (unsigned)(OFF_WS + (par * NR + q) * W_SLAB) + (row + i * 4) * NS + col, 0, AUX_SYS);
                WAIT_VM0();
                __syncthreads();
                if (tid == 0) st_sys((unsigned*)(p.ws[q] + OFF_FS) + rank * 64 + 32 + pi, call);
            }
        }
        const int wm = wave >> 2, wn = wave & 3;
        const float scale = p.sx[0] * p.sw[0];
        const unsigned char* const wsX = wsme + OFF_X;
        const unsigned char* const wsW = wsme + OFF_W + (size_t)par * NR * W_SLAB;

        unsigned xoff[2], woffR[2], woffO[2];
#pragma unroll
        for (int i = 0; i < 2; ++i) {
            const int r = wave * 16 + i * 8 + (lane >> 3);
            xoff[i] = r * KS + (((lane & 7) ^ sigma_x(r & 15)) << 4);
            const unsigned ch = (((lane & 7) ^ (((r >> 1) & 3) | (((r >> 5) & 1) << 2))) << 4);
            woffR[i] = r * NS + ch;
            woffO[i] = r * NG + ch;
        }
        const int g = lane >> 4, t16 = lane & 15;
        unsigned xa[2], wa[2];
#pragma unroll
        for (int h = 0; h < 2; ++h) xa[h] = (wm * 64 + t16) * BK + (((2 * g + h) ^ sigma_x(t16)) << 4);
        {
            const int F = ((t16 >> 2) & 3) | ((g & 1) << 2);
#pragma unroll
            for (int ni = 0; ni < 2; ++ni)
                wa[ni] = BM * BK + (32 * g + (t16 >> 1)) * BN + (((wn * 2 + ni) ^ F) << 4) + (t16 & 1) * 8;
        }
        const unsigned lds_base = (unsigned)(uintptr_t)(LDS_AS unsigned char*)smem;

        for (int idx = gi >> 3; idx < TILES_PER_GROUP; idx += GPX) {
            const int j = idx >> 4, ct = 2 * grp + (idx & 1), qq = (rank + 1 + ((idx >> 1) & 7)) & 7;
            const int R0 = qq * MS + j * BM;

            if (wave == 0) {
                const unsigned* f = nullptr;
                const int l7 = lane & 7;
                if (l7 < 7 && lane < 24) {
                    const int src = l7 + (l7 >= rank);
                    if (lane < 8)       f = (const unsigned*)(wsme + OFF_FW) + src;
                    else if (lane < 16) f = (const unsigned*)(wsme + OFF_FWL) + src;
                    else                f = (const unsigned*)(wsme + OFF_FX) + (j * 8 + qq) * 8 + src;
                }
                wait_flags(f, call, call, tmo, 1);
                __builtin_amdgcn_fence(__ATOMIC_ACQUIRE, "");
                WAIT_VM0();
            }
            __syncthreads();

            const unsigned char *xb, *wb;
            unsigned wstep, wo0, wo1;
            auto slab = [&](int ps) {
                const bool own = (ps == rank);
                xb = (own ? p.x : wsX + (size_t)ps * X_SLAB) + (size_t)R0 * KS;
                wb = (own ? p.w + rank * NS : wsW + (size_t)ps * W_SLAB) + ct * BN;
                wstep = own ? BK * NG : BK * NS;
                wo0 = own ? woffO[0] : woffR[0];
                wo1 = own ? woffO[1] : woffR[1];
            };
            auto dma = [&](unsigned lds, unsigned voff, const unsigned char* gbase) {
                asm volatile("s_mov_b32 m0, %0\n\ts_nop 0\n\tglobal_load_lds_dwordx4 %1, %2" :: "s"(lds), "v"(voff), "s"(gbase) : "memory");
            };
            auto stage = [&](int kk, int buf) {
                const unsigned lb = lds_base + buf * STAGE_BYTES + wave * 2048;
                const unsigned char* xk = xb + kk * BK;
                const unsigned char* wk = wb + kk * wstep;
                dma(lb, xoff[0], xk);
                dma(lb + 1024, xoff[1], xk);
                dma(lb + BM * BK, wo0, wk);
                dma(lb + BM * BK + 1024, wo1, wk);
            };

#define RD_X(R0, R1, A, OFF) "ds_read_b128 v[" #R0 ":" #R1 "], " A " offset:" #OFF "\n\t"
#define RD_W(R0, R1, A, OFF) "ds_read_b64_tr_b8 v[" #R0 ":" #R1 "], " A " offset:" #OFF "\n\t"
#define MM(C0, C1, W0, W1, X0, X1) "v_mfma_f32_16x16x128_f8f6f4 v[" #C0 ":" #C1 "], v[" #W0 ":" #W1 "], v[" #X0 ":" #X1 "], v[" #C0 ":" #C1 "] cbsz:1\n\t"
#define CLOB_ALL "v96","v97","v98","v99","v100","v101","v102","v103","v104","v105","v106","v107","v108","v109","v110","v111", \
               "v112","v113","v114","v115","v116","v117","v118","v119","v120","v121","v122","v123","v124","v125","v126","v127", \
               "v128","v129","v130","v131","v132","v133","v134","v135","v136","v137","v138","v139","v140","v141","v142","v143", \
               "v144","v145","v146","v147","v148","v149","v150","v151","v152","v153","v154","v155","v156","v157","v158","v159", \
               "v160","v161","v162","v163","v164","v165","v166","v167","v168","v169","v170","v171","v172","v173","v174","v175", \
               "v176","v177","v178","v179","v180","v181","v182","v183","v184","v185","v186","v187","v188","v189","v190","v191", \
               "v192","v193","v194","v195","v196","v197","v198","v199","v200","v201","v202","v203","v204","v205","v206","v207", \
               "v208","v209","v210","v211","v212","v213","v214","v215","v216","v217","v218","v219","v220","v221","v222","v223"
#define READ_A1 RD_W(160,161,"%2",0) RD_W(162,163,"%2",1024) RD_W(164,165,"%2",2048) RD_W(166,167,"%2",3072) \
                RD_X(128,131,"%0",0) RD_X(132,135,"%1",0) RD_X(136,139,"%0",2048) RD_X(140,143,"%1",2048)
#define READ_A2 RD_X(144,147,"%0",4096) RD_X(148,151,"%1",4096) RD_X(152,155,"%0",6144) RD_X(156,159,"%1",6144) \
                RD_W(168,169,"%3",0) RD_W(170,171,"%3",1024) RD_W(172,173,"%3",2048) RD_W(174,175,"%3",3072)
#define ZERO4(R) "v_mov_b32 v" #R ", 0\n\t"
            auto lds_addr = [&](int b, unsigned& a0, unsigned& a1, unsigned& a2, unsigned& a3) {
                const unsigned lb = lds_base + b * STAGE_BYTES;
                a0 = lb + xa[0]; a1 = lb + xa[1]; a2 = lb + wa[0]; a3 = lb + wa[1];
            };
            static_assert(NSTAGE == 5, "the K loop below prefetches exactly one slab (4 steps) ahead");
            slab(0);
#pragma unroll
            for (int i = 0; i < 4; ++i) stage(i, i);
            asm volatile(
                ZERO4(96) ZERO4(97) ZERO4(98) ZERO4(99) ZERO4(100) ZERO4(101) ZERO4(102) ZERO4(103)
                ZERO4(104) ZERO4(105) ZERO4(106) ZERO4(107) ZERO4(108) ZERO4(109) ZERO4(110) ZERO4(111)
                ZERO4(112) ZERO4(113) ZERO4(114) ZERO4(115) ZERO4(116) ZERO4(117) ZERO4(118) ZERO4(119)
                ZERO4(120) ZERO4(121) ZERO4(122) ZERO4(123) ZERO4(124) ZERO4(125) ZERO4(126) ZERO4(127)
                ::: CLOB_ALL);
            asm volatile("s_waitcnt vmcnt(12)" ::: "memory");
            __builtin_amdgcn_s_barrier();
            {
                unsigned a0, a1, a2, a3;
                lds_addr(0, a0, a1, a2, a3);
                asm volatile(READ_A1 READ_A2 :: "v"(a0), "v"(a1), "v"(a2), "v"(a3) : CLOB_ALL, "memory");
            }
            int buf = 0;
#define STEP(WAITN, STG, KK, CX0,CX0e,CX1,CX1e,CX2,CX2e,CX3,CX3e,CW0,CW0e,CW1,CW1e, \
             W0,W1,W2,W3,W4,W5,W6,W7, X0,X1,X2,X3,X4,X5,X6,X7,X8,X9,XA,XB,XC,XD,XE,XF, V0,V1,V2,V3,V4,V5,V6,V7) \
            do { \
                int nb = buf + 1; if (nb == NSTAGE) nb = 0; \
                unsigned a0, a1, a2, a3; \
                lds_addr(nb, a0, a1, a2, a3); \
                asm volatile("s_waitcnt vmcnt(" #WAITN ")" ::: "memory"); \
                __builtin_amdgcn_s_barrier(); \
                asm volatile("s_waitcnt lgkmcnt(0)\n\t" \
                    MM(96,99, CW0, CW0e, CX0, CX0e) RD_W(W0,W1,"%2",0) RD_W(W2,W3,"%2",1024) \
                    MM(100,103, CW0, CW0e, CX1, CX1e) RD_W(W4,W5,"%2",2048) RD_W(W6,W7,"%2",3072) \
                    MM(104,107, CW0, CW0e, CX2, CX2e) RD_X(X0,X1,"%0",0) RD_X(X2,X3,"%1",0) \
                    MM(108,111, CW0, CW0e, CX3, CX3e) RD_X(X4,X5,"%0",2048) RD_X(X6,X7,"%1",2048) \
                    :: "v"(a0), "v"(a1), "v"(a2) : CLOB_ALL, "memory"); \
                if (STG) { int pb = buf - 1; if (pb < 0) pb += NSTAGE; stage(KK, pb); } \
                asm volatile( \
                    MM(112,115, CW1, CW1e, CX0, CX0e) RD_X(X8,X9,"%0",4096) RD_X(XA,XB,"%1",4096) \
                    MM(116,119, CW1, CW1e, CX1, CX1e) RD_X(XC,XD,"%0",6144) RD_X(XE,XF,"%1",6144) \
                    MM(120,123, CW1, CW1e, CX2, CX2e) RD_W(V0,V1,"%2",0) RD_W(V2,V3,"%2",1024) \
                    MM(124,127, CW1, CW1e, CX3, CX3e) RD_W(V4,V5,"%2",2048) RD_W(V6,V7,"%2",3072) \
                    :: "v"(a0), "v"(a1), "v"(a3) : CLOB_ALL, "memory"); \
                buf = nb; \
            } while (0)
#define STEP_A(WAITN, STG, KK) STEP(WAITN, STG, KK, 128,135,136,143,144,151,152,159,160,167,168,175, \
                     208,209,210,211,212,213,214,215, 176,179,180,183,184,187,188,191,192,195,196,199,200,203,204,207, \
                     216,217,218,219,220,221,222,223)
#define STEP_B(WAITN, STG, KK) STEP(WAITN, STG, KK, 176,183,184,191,192,199,200,207,208,215,216,223, \
                     160,161,162,163,164,165,166,167, 128,131,132,135,136,139,140,143,144,147,148,151,152,155,156,159, \
                     168,169,170,171,172,173,174,175)
            for (int ps = 0; ps < NR - 1; ++ps) {
                slab(ps + 1);
                STEP_A(8, 1, 0);
                STEP_B(8, 1, 1);
                STEP_A(8, 1, 2);
                STEP_B(8, 1, 3);
            }
            STEP_A(8, 0, 0);
            STEP_B(4, 0, 1);
            STEP_A(0, 0, 2);
            asm volatile("s_waitcnt lgkmcnt(0)\n\t"
                MM(96,99, 208,215, 176,183) MM(100,103, 208,215, 184,191) MM(104,107, 208,215, 192,199) MM(108,111, 208,215, 200,207)
                MM(112,115, 216,223, 176,183) MM(116,119, 216,223, 184,191) MM(120,123, 216,223, 192,199) MM(124,127, 216,223, 200,207)
                "s_nop 15\n\ts_nop 15"
                ::: CLOB_ALL, "memory");
            f32x4 acc[2][4];
#define GET4(T, R0, R1, R2, R3) do { float y0, y1, y2, y3; \
                asm volatile("v_mul_f32 %0, %4, v" #R0 "\n\tv_mul_f32 %1, %4, v" #R1 "\n\tv_mul_f32 %2, %4, v" #R2 "\n\tv_mul_f32 %3, %4, v" #R3 \
                    : "=&v"(y0), "=&v"(y1), "=&v"(y2), "=&v"(y3) : "v"(scale) : CLOB_ALL); \
                T = f32x4{y0, y1, y2, y3}; } while (0)
            GET4(acc[0][0], 96, 97, 98, 99);     GET4(acc[0][1], 100, 101, 102, 103);
            GET4(acc[0][2], 104, 105, 106, 107); GET4(acc[0][3], 108, 109, 110, 111);
            GET4(acc[1][0], 112, 113, 114, 115); GET4(acc[1][1], 116, 117, 118, 119);
            GET4(acc[1][2], 120, 121, 122, 123); GET4(acc[1][3], 124, 125, 126, 127);

            const bool bad = __hip_atomic_load(tmo, __ATOMIC_RELAXED, __HIP_MEMORY_SCOPE_AGENT) == call;
            if (bad) {
                const float qnan = __builtin_bit_cast(float, 0x7fc00000u);
#pragma unroll
                for (int a = 0; a < 2; ++a)
#pragma unroll
                    for (int b = 0; b < 4; ++b) acc[a][b] = f32x4{qnan, qnan, qnan, qnan};
            }
            if (qq == rank) {
                const __amdgpu_buffer_rsrc_t ro = make_rsrc(p.out, (unsigned)((size_t)MS * NG * 4));
                const unsigned dst0 = ((unsigned)(j * BM + wm * 64 + t16) * NG + rank * NS + ct * BN + wn * 32 + g * 4) * 4;
#pragma unroll
                for (int mi = 0; mi < 4; ++mi)
#pragma unroll
                    for (int ni = 0; ni < 2; ++ni) {
                        f32x4 y;
#pragma unroll
                        for (int e = 0; e < 4; ++e) { const float v = acc[ni][mi][e]; y[e] = v < 0.f ? 0.f : v; }
                        __builtin_amdgcn_raw_buffer_store_b128(__builtin_bit_cast(u32x4, y), ro, dst0 + (mi * 16 * NG + ni * 16) * 4, 0, OUT_AUX);
                    }
            } else {
                __syncthreads();
                unsigned char* const wl = smem + wave * BLK_BYTES;
                const unsigned val0 = lds_base + wave * BLK_BYTES + 256;
                unsigned posb = val0, word = 0;
#pragma unroll
                for (int ni = 0; ni < 2; ++ni)
#pragma unroll
                    for (int mi = 0; mi < 4; ++mi)
#pragma unroll
                        for (int e = 0; e < 4; ++e) {
                            unsigned t, c;
                            unsigned long long co;
                            asm volatile(
                                "v_cmp_nge_f32 vcc, 0, %[v]\n\t"
                                "s_nop 1\n\t"
                                "v_addc_co_u32 %[w], %[co], %[w], %[w], vcc\n\t"
                                "v_mbcnt_lo_u32_b32 %[t], vcc_lo, 0\n\t"
                                "v_mbcnt_hi_u32_b32 %[t], vcc_hi, %[t]\n\t"
                                "v_lshl_add_u32 %[t], %[t], 2, %[p]\n\t"
                                "s_mov_b64 exec, vcc\n\t"
                                "ds_write_b32 %[t], %[v]\n\t"
                                "s_mov_b64 exec, -1\n\t"
                                "s_bcnt1_i32_b64 %[c], vcc\n\t"
                                "s_lshl2_add_u32 %[p], %[c], %[p]\n\t"
                                "s_nop 0"
                                : [t] "=&v"(t), [c] "=&s"(c), [co] "=&s"(co), [w] "+v"(word), [p] "+s"(posb)
                                : [v] "v"(acc[ni][mi][e])
                                : "vcc", "scc", "memory");
                        }
                const unsigned pos = (posb - val0) >> 2;
                *(unsigned*)(wl + lane * 4) = word;
                asm volatile("s_waitcnt lgkmcnt(0)" ::: "memory");
                const unsigned bytes = 256u + pos * 4u;
                const __amdgpu_buffer_rsrc_t rs = make_rsrc(wsme, (unsigned)WS_USED);
                const unsigned blk = (j * NCT + ct) * 8 + wave;
                const unsigned dst0 = (unsigned)(OFF_O + (size_t)qq * O_SLAB) + blk * BLK_BYTES;
#pragma unroll
                for (int c = 0; c < 9; ++c) {
                    const unsigned off = c * 1024 + lane * 16;
                    if (off < bytes)
                        __builtin_amdgcn_raw_buffer_store_b128(*(const u32x4*)(wl + off), rs, off, dst0, AUX_SYS);
                }
                WAIT_VM0();
                if (lane == 0)
                    __hip_atomic_store((unsigned long long*)(p.ws[qq] + OFF_FO) + rank * 512 + blk,
                                       ((unsigned long long)call << 32) | pos, __ATOMIC_RELAXED, __HIP_MEMORY_SCOPE_SYSTEM);
            }
            __syncthreads();
        }
    }
}

}

extern "C" void kernel_launch(int rank, int nranks, ncclComm_t comm,
                              hipStream_t stream,
                              void* const* d_in, const int* in_sizes, int n_in,
                              void* d_out, int out_size,
                              const int* in_dtypes, int out_dtype,
                              void* const* d_out_all,
                              void* const* d_ws_all, size_t ws_bytes) {
    (void)comm; (void)in_dtypes; (void)out_dtype; (void)d_out_all;
    if (nranks != NR || n_in < 4 || rank < 0 || rank >= NR || ws_bytes < WS_USED ||
        in_sizes[0] != MG * KS || in_sizes[1] != KS * NG || out_size != MS * NG) {
        fprintf(stderr, "kernel_launch: unexpected problem shape\n");
        return;
    }
    Params p;
    p.x = (const unsigned char*)d_in[0];
    p.w = (const unsigned char*)d_in[1];
    p.sx = (const float*)d_in[2];
    p.sw = (const float*)d_in[3];
    p.out = (float*)d_out;
    for (int i = 0; i < NR; ++i) p.ws[i] = (unsigned char*)d_ws_all[i];
    p.rank = rank;
    hipLaunchKernelGGL(gemm_rs_fused, dim3(NWG), dim3(512), 0, stream, p);
    HIP_CHECK(hipGetLastError());
}
```

```cpp
#include <hip/hip_runtime.h>
#include <rccl/rccl.h>
#include <cstdio>
#include <cstdint>

#define HIP_CHECK(expr) do { hipError_t _e = (expr); if (_e != hipSuccess) { \
    fprintf(stderr, "HIP error %s at %s:%d\n", hipGetErrorString(_e), __FILE__, __LINE__); \
    return; } } while (0)

namespace {

constexpr int NR = 8;
constexpr int MG = 4096, KG = 4096, NG = 16384;
constexpr int KS = KG / NR;
constexpr int MS = MG / NR;
constexpr int NS = NG / NR;
constexpr int BM = 128, BN = 128, BK = 128;
constexpr int NCHUNK = MS / BM;
constexpr int NCT = NS / BN;
constexpr int NCOMM = 56, NWG = 256;
constexpr int GPX = 25;
constexpr int NSTAGE = 5;
#define PULL_DEPTH 2
#define PULL_DEPTH_M1 1
#define OUT_AUX 16
constexpr int TILES_PER_GROUP = NCHUNK * NR * 2;

constexpr unsigned U_WP = 64;
constexpr unsigned U_W = 256;
constexpr unsigned U_XP = 16;
constexpr unsigned U_XEND = U_W + NCHUNK * NR * U_XP;
constexpr unsigned U_PULL = NCHUNK * NCT * 8;
constexpr unsigned U_ALL = U_XEND + U_PULL;
constexpr int WPIECES = (256 - U_WP) / 16;
constexpr unsigned LINK_WAVES = 64;
#define TAIL_UNITS 64

constexpr size_t MiB = 1ull << 20;
constexpr size_t OFF_FW  = 0;
constexpr size_t OFF_FWL = 128;
constexpr size_t OFF_FD  = 512;
constexpr size_t OFF_FX  = 1536;
constexpr size_t OFF_TK  = 3072;
constexpr size_t OFF_FS  = 4096;
constexpr size_t OFF_TMO = 24576;
constexpr size_t OFF_CNT = 32768;
constexpr size_t OFF_PC  = 65536;
constexpr size_t OFF_FO  = 131072;
constexpr size_t X_SLAB  = 2 * MiB;
constexpr size_t OFF_XS  = 1 * MiB;
constexpr size_t OFF_X   = OFF_XS + 2 * X_SLAB;
constexpr size_t OFF_W   = OFF_X + NR * X_SLAB;
constexpr size_t W_SLAB  = 1 * MiB;
constexpr unsigned BLK_BYTES = 256 + 8192;
constexpr size_t OFF_WS  = OFF_W + 2 * NR * W_SLAB;
constexpr size_t OFF_O   = OFF_WS + 2 * NR * W_SLAB;
constexpr size_t O_SLAB  = 5 * MiB;
constexpr size_t WS_USED = OFF_O + NR * O_SLAB;
static_assert(512 * BLK_BYTES <= O_SLAB, "slab");

constexpr unsigned long long DEADLINE_TICKS = 150000000ull;

constexpr int STAGE_BYTES = BM * BK + BK * BN;
constexpr int LDS_BYTES = NSTAGE * STAGE_BYTES;

typedef unsigned u32x4 __attribute__((ext_vector_type(4)));
typedef float f32x4 __attribute__((ext_vector_type(4)));
#define LDS_AS __attribute__((address_space(3)))

struct Params {
    const unsigned char* x;
    const unsigned char* w;
    const float* sx;
    const float* sw;
    float* out;
    unsigned char* ws[NR];
    int rank;
};

constexpr int AUX_AGENT = 16;
constexpr int AUX_SYS = 17;

__device__ __forceinline__ __amdgpu_buffer_rsrc_t make_rsrc(const void* base, unsigned bytes) {
    return __builtin_amdgcn_make_buffer_rsrc(const_cast<void*>(base), 0, bytes, 0x00020000);
}
__device__ __forceinline__ unsigned ld_sys(const unsigned* p) {
    return __hip_atomic_load(p, __ATOMIC_RELAXED, __HIP_MEMORY_SCOPE_SYSTEM);
}
__device__ __forceinline__ void st_sys(unsigned* p, unsigned v) {
    __hip_atomic_store(p, v, __ATOMIC_RELAXED, __HIP_MEMORY_SCOPE_SYSTEM);
}
#define WAIT_VM0() asm volatile("s_waitcnt vmcnt(0)" ::: "memory")
#define DRAW_ASYNC(REG, ctr) do { if (lane == 0) \
    asm volatile("global_atomic_add " REG ", %0, %1, off sc0" :: "v"(ctr), "v"(1u) : REG, "memory"); } while (0)
#define STR_(x) #x
#define STR(x) STR_(x)
#define DRAW_GET(REG, N, dst) \
    asm volatile("s_waitcnt vmcnt(" STR(N) ")\n\tv_readfirstlane_b32 %0, " REG "\n\ts_nop 3" : "=s"(dst) :: REG, "memory")

__device__ __forceinline__ bool wait_flags(const unsigned* fp, unsigned want, unsigned call, unsigned* tmo, int sleep_long) {
    const unsigned long long t0 = __builtin_amdgcn_s_memrealtime();
    for (;;) {
        const unsigned v = fp ? ld_sys(fp) : want;
        const unsigned t = __hip_atomic_load(tmo, __ATOMIC_RELAXED, __HIP_MEMORY_SCOPE_AGENT);
        if (t == call) return false;
        if (__all((int)(v - want) >= 0)) return true;
        if (__builtin_amdgcn_s_memrealtime() - t0 > DEADLINE_TICKS) {
            __hip_atomic_store(tmo, call, __ATOMIC_RELAXED, __HIP_MEMORY_SCOPE_AGENT);
            return false;
        }
        if (sleep_long) __builtin_amdgcn_s_sleep(16); else __builtin_amdgcn_s_sleep(2);
    }
}

__device__ __forceinline__ unsigned long long wait_staged(const unsigned* fs, unsigned need, unsigned call, unsigned* tmo, int lane) {
    const unsigned long long t0 = __builtin_amdgcn_s_memrealtime();
    for (;;) {
        const unsigned v = ld_sys(fs + lane);
        const unsigned t = __hip_atomic_load(tmo, __ATOMIC_RELAXED, __HIP_MEMORY_SCOPE_AGENT);
        const unsigned long long m = __builtin_amdgcn_ballot_w64((int)(v - call) >= 0);
        if (t == call) return ~0ull;
        if ((m >> need) & 1ull) return m;
        if (__builtin_amdgcn_s_memrealtime() - t0 > DEADLINE_TICKS) {
            __hip_atomic_store(tmo, call, __ATOMIC_RELAXED, __HIP_MEMORY_SCOPE_AGENT);
            return ~0ull;
        }
        __builtin_amdgcn_s_sleep(2);
    }
}
__device__ __forceinline__ bool wait_flag64(const unsigned long long* fp, unsigned call, unsigned* tmo, unsigned& payload) {
    const unsigned long long t0 = __builtin_amdgcn_s_memrealtime();
    for (;;) {
        const unsigned long long v = __hip_atomic_load(fp, __ATOMIC_RELAXED, __HIP_MEMORY_SCOPE_SYSTEM);
        const unsigned t = __hip_atomic_load(tmo, __ATOMIC_RELAXED, __HIP_MEMORY_SCOPE_AGENT);
        const unsigned hi = __builtin_amdgcn_readfirstlane((unsigned)(v >> 32));
        payload = __builtin_amdgcn_readfirstlane((unsigned)v);
        if (t == call) return false;
        if ((int)(hi - call) >= 0) return true;
        if (__builtin_amdgcn_s_memrealtime() - t0 > DEADLINE_TICKS) {
            __hip_atomic_store(tmo, call, __ATOMIC_RELAXED, __HIP_MEMORY_SCOPE_AGENT);
            return false;
        }
        __builtin_amdgcn_s_sleep(2);
    }
}
__device__ __forceinline__ int sigma_x(int r) { return ((r >> 1) & 1) | (((r >> 3) & 1) * 6); }

__global__ __launch_bounds__(512, 2) void gemm_rs_fused(Params p) {
    __shared__ __attribute__((aligned(16))) unsigned char smem[LDS_BYTES];
    const int tid = threadIdx.x;
    const int lane = tid & 63;
    const int wave = __builtin_amdgcn_readfirstlane(tid >> 6);
    const int rank = p.rank;
    unsigned char* const wsme = p.ws[rank];
    unsigned* const tmo = (unsigned*)(wsme + OFF_TMO);
    unsigned* const cntp = (unsigned*)(wsme + OFF_CNT + (size_t)blockIdx.x * 64);

    const unsigned cnt_raw = __hip_atomic_load(cntp, __ATOMIC_RELAXED, __HIP_MEMORY_SCOPE_AGENT);

    if (blockIdx.x < NCOMM) {
        const int li = blockIdx.x >> 3;
        const unsigned q = (unsigned)(rank + 1 + li) & 7u;
        unsigned char* const wsq = p.ws[q];
        unsigned* const tkp = (unsigned*)(wsme + OFF_TK + li * 128);
        const __amdgpu_buffer_rsrc_t rw = make_rsrc(p.w, (unsigned)((size_t)KS * NG));
        const __amdgpu_buffer_rsrc_t rq = make_rsrc(wsq, (unsigned)WS_USED);
        const __amdgpu_buffer_rsrc_t rm = make_rsrc(wsme, (unsigned)WS_USED);
        const __amdgpu_buffer_rsrc_t ro = make_rsrc(p.out, (unsigned)((size_t)MS * NG * 4));

        unsigned u = (blockIdx.x & 7) * 8 + wave;
        DRAW_ASYNC("v250", tkp);
        const unsigned fd_early = ld_sys((const unsigned*)(wsme + OFF_FD) + q * 32);
        const unsigned l16 = lane * 16;
        u32x4 v[4];
#pragma unroll
        for (int i = 0; i < 4; ++i)
            v[i] = __builtin_amdgcn_raw_buffer_load_b128(rw, l16 + (i & 1) * 1024, (2 * u + (i >> 1)) * NG + q * NS, 0);
        const unsigned call = __builtin_amdgcn_readfirstlane(cnt_raw) + 1u;
        const unsigned par = call & 1u;
        const unsigned tk_base = (call - 1u) * (U_ALL - U_WP + LINK_WAVES) - U_WP;
        if ((int)(__builtin_amdgcn_readfirstlane(fd_early) - (call - 1u) * LINK_WAVES) < 0)
            wait_flags(lane == 0 ? (const unsigned*)(wsme + OFF_FD) + q * 32 : nullptr, (call - 1u) * LINK_WAVES, call, tmo, 0);

        unsigned* const pcb = (unsigned*)(wsme + OFF_PC + li * 4096);
        auto publish = [&](unsigned pc, unsigned old) {
            if (lane == 0) {
                if (pc == 34u) {
                    if (old + 1u == call * LINK_WAVES) st_sys((unsigned*)(wsq + OFF_FW) + rank, call);
                } else if (pc == 0u) {
                    if (old + 1u == call * (U_W - U_WP))
                        __hip_atomic_store((unsigned*)(wsme + OFF_FWL) + q, call, __ATOMIC_RELAXED, __HIP_MEMORY_SCOPE_AGENT);
                } else if (old + 1u == call * U_XP) {
                    __hip_atomic_store((unsigned*)(wsme + OFF_FX) + (pc - 1u) * 8 + q, call, __ATOMIC_RELAXED, __HIP_MEMORY_SCOPE_AGENT);
                }
            }
        };
#pragma unroll
        for (int i = 0; i < 4; ++i)
            __builtin_amdgcn_raw_buffer_store_b128(v[i], rq, l16 + i * 1024, (unsigned)(OFF_W + (par * NR + rank) * W_SLAB) + u * 4096, AUX_SYS);
        __syncthreads();
        if (tid == 0) __hip_atomic_store(cntp, call, __ATOMIC_RELAXED, __HIP_MEMORY_SCOPE_AGENT);
        unsigned pc_prev = 34u;
        {
            unsigned tn;
            DRAW_GET("v250", 5, tn);
            u = tn - tk_base;
            if (u < U_ALL) DRAW_ASYNC("v250", tkp);
        }

        unsigned long long staged = 0;
        const unsigned long long* const fo = (const unsigned long long*)(wsme + OFF_FO) + q * 512;
        bool pre = false;
        while (u < U_XEND) {
            unsigned src0, dst0, pc, need;
            if (u < U_W) {
                pc = 0;
                need = 32 + ((u - U_WP) >> 4);
                src0 = (unsigned)(OFF_WS + (par * NR + rank) * W_SLAB) + u * 4096;
                dst0 = (unsigned)(OFF_W + (par * NR + q) * W_SLAB) + u * 4096;
            } else {
                const unsigned t = u - U_W, j = t >> 7, qq = (rank + 1 + (t >> 4)) & 7;
                const unsigned off = (qq * MS + j * BM) * KS + (t & 15) * 4096;
                pc = 1 + j * 8 + qq;
                need = j * 8 + qq;
                src0 = (unsigned)(OFF_XS + par * X_SLAB) + off;
                dst0 = (unsigned)(OFF_X + q * X_SLAB) + off;
            }
            if (!((staged >> need) & 1ull)) staged = wait_staged((const unsigned*)(wsme + OFF_FS) + q * 64, need, call, tmo, lane);
            unsigned un = 0, old;
            v[0] = __builtin_amdgcn_raw_buffer_load_b128(rq, l16, src0, AUX_SYS);
            DRAW_GET("v250", 1, un);
            un -= tk_base;
            if (un < U_ALL) DRAW_ASYNC("v250", tkp);
            DRAW_ASYNC("v251", pcb + pc_prev * 16);
            if (un >= U_XEND && un < U_ALL)
                asm volatile("global_load_dwordx2 v[252:253], %0, off sc0 sc1" :: "v"(fo + (un - U_XEND)) : "v252", "v253", "memory");
            __builtin_amdgcn_raw_buffer_store_b128(v[0], rm, l16, dst0, AUX_AGENT);
            DRAW_GET("v251", 1, old);
            publish(pc_prev, old);
#pragma unroll
            for (int f = 1; f < 4; ++f) {
                v[f] = __builtin_amdgcn_raw_buffer_load_b128(rq, l16 + f * 1024, src0, AUX_SYS);
                __builtin_amdgcn_raw_buffer_store_b128(v[f], rm, l16 + f * 1024, dst0, AUX_AGENT);
            }
            pre = un >= U_XEND && un < U_ALL;
            pc_prev = pc;
            u = un;
        }
        {
            unsigned old;
            WAIT_VM0();
            DRAW_ASYNC("v251", pcb + pc_prev * 16);
            DRAW_GET("v251", 0, old);
            publish(pc_prev, old);
        }

        const unsigned olane = ((lane & 15) * NG + (lane >> 4) * 4) * 4;
        unsigned char* const wl = smem + wave * BLK_BYTES;
        const unsigned lds_wl = (unsigned)(uintptr_t)(LDS_AS unsigned char*)smem + wave * BLK_BYTES;
#define CLOB_Y "v96","v97","v98","v99","v100","v101","v102","v103","v104","v105","v106","v107","v108","v109","v110","v111", \
               "v112","v113","v114","v115","v116","v117","v118","v119","v120","v121","v122","v123","v124","v125","v126","v127","v128"
        bool ready = false;
        unsigned cnt = 0;
        if (pre) {
            unsigned fhi;
            asm volatile("v_readfirstlane_b32 %0, v252\n\tv_readfirstlane_b32 %1, v253\n\ts_nop 3"
                         : "=s"(cnt), "=s"(fhi) :: "v252", "v253", "memory");
            ready = (int)(fhi - call) >= 0;
        }
        while (u < U_ALL) {
            const unsigned blk = u - U_XEND;
            const unsigned j = blk >> 7, ct = (blk >> 3) & 15, w = blk & 7;
            bool ok = true;
            if (!ready) {
                ok = wait_flag64(fo + blk, call, tmo, cnt);
                asm volatile("" ::: "memory");
            }
            const unsigned dst0 = ((j * BM + (w >> 2) * 64) * NG + q * NS + ct * BN + (w & 3) * 32) * 4;
            unsigned un = 0;
            if (ok) {
                if (cnt > 2048u) cnt = 2048u;
                const unsigned bytes = 256u + cnt * 4u;
                const unsigned char* src = wsq + OFF_O + (size_t)rank * O_SLAB + (size_t)blk * BLK_BYTES + l16;
                for (unsigned c = 0; c * 1024u < bytes; ++c) {
                    if (c * 1024u + l16 < bytes)
                        __builtin_amdgcn_global_load_lds((const unsigned*)(src + c * 1024u), (LDS_AS unsigned*)(wl + c * 1024u), 16, 0, AUX_SYS);
                    if (c == 0) {
                        DRAW_GET("v250", 1, un);
                        un -= tk_base;
                        if (un < U_ALL) {
                            DRAW_ASYNC("v250", tkp);
                            asm volatile("global_load_dwordx2 v[252:253], %0, off sc0 sc1" :: "v"(fo + (un - U_XEND)) : "v252", "v253", "memory");
                        }
                    }
                    if (c >= PULL_DEPTH - 1 && u < U_ALL - TAIL_UNITS) asm volatile("s_waitcnt vmcnt(" STR(PULL_DEPTH_M1) ")" ::: "memory");
                }
                WAIT_VM0();
                {
                    unsigned word = *(const unsigned*)(wl + lane * 4);
                    unsigned posb = lds_wl + 256;
                    asm volatile(
                        "v_mov_b32 v96, 0\n\tv_mov_b32 v97, 0\n\tv_mov_b32 v98, 0\n\tv_mov_b32 v99, 0\n\t"
                        "v_mov_b32 v100, 0\n\tv_mov_b32 v101, 0\n\tv_mov_b32 v102, 0\n\tv_mov_b32 v103, 0\n\t"
                        "v_mov_b32 v104, 0\n\tv_mov_b32 v105, 0\n\tv_mov_b32 v106, 0\n\tv_mov_b32 v107, 0\n\t"
                        "v_mov_b32 v108, 0\n\tv_mov_b32 v109, 0\n\tv_mov_b32 v110, 0\n\tv_mov_b32 v111, 0\n\t"
                        "v_mov_b32 v112, 0\n\tv_mov_b32 v113, 0\n\tv_mov_b32 v114, 0\n\tv_mov_b32 v115, 0\n\t"
                        "v_mov_b32 v116, 0\n\tv_mov_b32 v117, 0\n\tv_mov_b32 v118, 0\n\tv_mov_b32 v119, 0\n\t"
                        "v_mov_b32 v120, 0\n\tv_mov_b32 v121, 0\n\tv_mov_b32 v122, 0\n\tv_mov_b32 v123, 0\n\t"
                        "v_mov_b32 v124, 0\n\tv_mov_b32 v125, 0\n\tv_mov_b32 v126, 0\n\tv_mov_b32 v127, 0"
                        ::: CLOB_Y);
#define UNP(R) asm volatile( \
                        "v_cmp_gt_i32 vcc, 0, %[w]\n\t" \
                        "v_add_u32 %[w], %[w], %[w]\n\t" \
                        "s_nop 0\n\t" \
                        "v_mbcnt_lo_u32_b32 v128, vcc_lo, 0\n\t" \
                        "v_mbcnt_hi_u32_b32 v128, vcc_hi, v128\n\t" \
                        "v_lshl_add_u32 v128, v128, 2, %[p]\n\t" \
                        "v_min_u32 v128, v128, %[lim]\n\t" \
                        "s_mov_b64 exec, vcc\n\t" \
                        "ds_read_b32 v" #R ", v128\n\t" \
                        "s_mov_b64 exec, -1\n\t" \
                        "s_bcnt1_i32_b64 %[c], vcc\n\t" \
                        "s_lshl2_add_u32 %[p], %[c], %[p]\n\t" \
                        "s_nop 0" \
                        : [c] "=&s"(cc), [w] "+v"(word), [p] "+s"(posb) : [lim] "v"(lim) : "vcc", "scc", "memory", CLOB_Y)
                    unsigned cc;
                    const unsigned lim = lds_wl + BLK_BYTES - 4;
                    UNP(96);  UNP(97);  UNP(98);  UNP(99);  UNP(100); UNP(101); UNP(102); UNP(103);
                    UNP(104); UNP(105); UNP(106); UNP(107); UNP(108); UNP(109); UNP(110); UNP(111);
                    UNP(112); UNP(113); UNP(114); UNP(115); UNP(116); UNP(117); UNP(118); UNP(119);
                    UNP(120); UNP(121); UNP(122); UNP(123); UNP(124); UNP(125); UNP(126); UNP(127);
                    unsigned char* const ob = (unsigned char*)p.out + dst0 + olane;
#define OST(R0, R1, F) asm volatile("global_store_dwordx4 %0, v[" #R0 ":" #R1 "], off sc1\n\ts_nop 1" \
                        :: "v"(ob + ((F) & 3) * (16 * NG * 4) + ((F) >> 2) * 64) : "memory", CLOB_Y)
                    asm volatile("s_waitcnt lgkmcnt(0)" ::: "memory", CLOB_Y);
                    OST(96, 99, 0);   OST(112, 115, 4); OST(100, 103, 1); OST(116, 119, 5);
                    OST(104, 107, 2); OST(120, 123, 6); OST(108, 111, 3); OST(124, 127, 7);
                }
                ready = false;
                if (un < U_ALL) {
                    unsigned fhi;
                    asm volatile("s_waitcnt vmcnt(8)\n\tv_readfirstlane_b32 %0, v252\n\tv_readfirstlane_b32 %1, v253\n\ts_nop 3"
                                 : "=s"(cnt), "=s"(fhi) :: "v252", "v253", "memory");
                    ready = (int)(fhi - call) >= 0;
                }
            } else {
#pragma unroll
                for (int f = 0; f < 8; ++f)
                    __builtin_amdgcn_raw_buffer_store_b128(u32x4{0x7fc00000u, 0x7fc00000u, 0x7fc00000u, 0x7fc00000u}, ro,
                                                           olane + (f >> 2) * 64, dst0 + (f & 3) * 16 * NG * 4, OUT_AUX);
                DRAW_GET("v250", 0, un);
                un -= tk_base;
                if (un < U_ALL) DRAW_ASYNC("v250", tkp);
                ready = false;
            }
            u = un;
        }
        if (lane == 0)
            (void)__hip_atomic_fetch_add((unsigned*)(wsq + OFF_FD) + rank * 32, 1u, __ATOMIC_RELAXED, __HIP_MEMORY_SCOPE_SYSTEM);
    } else {
        unsigned call = __builtin_amdgcn_readfirstlane(cnt_raw) + 1u;
        const unsigned par = call & 1u;
        __syncthreads();
        if (tid == 0) __hip_atomic_store(cntp, call, __ATOMIC_RELAXED, __HIP_MEMORY_SCOPE_AGENT);
        const int gi = blockIdx.x - NCOMM;
        const int grp = gi & 7;
        if (gi < NCHUNK * NR + (NR - 1) * WPIECES) {
            const __amdgpu_buffer_rsrc_t rm = make_rsrc(wsme, (unsigned)WS_USED);
            u32x4 v[8];
            if (gi >= (NR - 1) * WPIECES) {
                const int xi = gi - (NR - 1) * WPIECES;
                const unsigned off = ((xi & 7) * MS + (xi >> 3) * BM) * KS + tid * 16;
                const __amdgpu_buffer_rsrc_t rx = make_rsrc(p.x, (unsigned)((size_t)MG * KS));
#pragma unroll
                for (int i = 0; i < 8; ++i) v[i] = __builtin_amdgcn_raw_buffer_load_b128(rx, off + i * 8192, 0, 0);
#pragma unroll
                for (int i = 0; i < 8; ++i)
                    __builtin_amdgcn_raw_buffer_store_b128(v[i], rm, (unsigned)(OFF_XS + par * X_SLAB) + off + i * 8192, 0, AUX_SYS);
                WAIT_VM0();
                __syncthreads();
                if (tid < NR && tid != rank) st_sys((unsigned*)(p.ws[tid] + OFF_FS) + rank * 64 + xi, call);
            } else {
                const int pi = gi / (NR - 1), d = gi % (NR - 1);
                const unsigned q = (unsigned)(rank + 1 + d) & 7u;
                const unsigned row = 2 * U_WP + pi * 32 + (tid >> 7), col = (tid & 127) * 16;
                const __amdgpu_buffer_rsrc_t rw = make_rsrc(p.w, (unsigned)((size_t)KS * NG));
#pragma unroll
                for (int i = 0; i < 8; ++i) v[i] = __builtin_amdgcn_raw_buffer_load_b128(rw, (row + i * 4) * NG + q * NS + col, 0, 0);
#pragma unroll
                for (int i = 0; i < 8; ++i)
                    __builtin_amdgcn_raw_buffer_store_b128(v[i], rm, (unsigned)(OFF_WS + (par * NR + q) * W_SLAB) + (row + i * 4) * NS + col, 0, AUX_SYS);
                WAIT_VM0();
                __syncthreads();
                if (tid == 0) st_sys((unsigned*)(p.ws[q] + OFF_FS) + rank * 64 + 32 + pi, call);
            }
        }
        const int wm = wave >> 2, wn = wave & 3;
        const float scale = p.sx[0] * p.sw[0];
        const unsigned char* const wsX = wsme + OFF_X;
        const unsigned char* const wsW = wsme + OFF_W + (size_t)par * NR * W_SLAB;

        unsigned xoff[2], woffR[2], woffO[2];
#pragma unroll
        for (int i = 0; i < 2; ++i) {
            const int r = wave * 16 + i * 8 + (lane >> 3);
            xoff[i] = r * KS + (((lane & 7) ^ sigma_x(r & 15)) << 4);
            const unsigned ch = (((lane & 7) ^ (((r >> 1) & 3) | (((r >> 5) & 1) << 2))) << 4);
            woffR[i] = r * NS + ch;
            woffO[i] = r * NG + ch;
        }
        const int g = lane >> 4, t16 = lane & 15;
        unsigned xa[2], wa[2];
#pragma unroll
        for (int h = 0; h < 2; ++h) xa[h] = (wm * 64 + t16) * BK + (((2 * g + h) ^ sigma_x(t16)) << 4);
        {
            const int F = ((t16 >> 2) & 3) | ((g & 1) << 2);
#pragma unroll
            for (int ni = 0; ni < 2; ++ni)
                wa[ni] = BM * BK + (32 * g + (t16 >> 1)) * BN + (((wn * 2 + ni) ^ F) << 4) + (t16 & 1) * 8;
        }
        const unsigned lds_base = (unsigned)(uintptr_t)(LDS_AS unsigned char*)smem;

        for (int idx = gi >> 3; idx < TILES_PER_GROUP; idx += GPX) {
            const int j = idx >> 4, ct = 2 * grp + (idx & 1), qq = (rank + 1 + ((idx >> 1) & 7)) & 7;
            const int R0 = qq * MS + j * BM;

            if (wave == 0) {
                const unsigned* f = nullptr;
                const int l7 = lane & 7;
                if (l7 < 7 && lane < 24) {
                    const int src = l7 + (l7 >= rank);
                    if (lane < 8)       f = (const unsigned*)(wsme + OFF_FW) + src;
                    else if (lane < 16) f = (const unsigned*)(wsme + OFF_FWL) + src;
                    else                f = (const unsigned*)(wsme + OFF_FX) + (j * 8 + qq) * 8 + src;
                }
                wait_flags(f, call, call, tmo, 1);
                __builtin_amdgcn_fence(__ATOMIC_ACQUIRE, "");
                WAIT_VM0();
            }
            __syncthreads();

            const unsigned char *xb, *wb;
            unsigned wstep, wo0, wo1;
            auto slab = [&](int ps) {
                const bool own = (ps == rank);
                xb = (own ? p.x : wsX + (size_t)ps * X_SLAB) + (size_t)R0 * KS;
                wb = (own ? p.w + rank * NS : wsW + (size_t)ps * W_SLAB) + ct * BN;
                wstep = own ? BK * NG : BK * NS;
                wo0 = own ? woffO[0] : woffR[0];
                wo1 = own ? woffO[1] : woffR[1];
            };
            auto dma = [&](unsigned lds, unsigned voff, const unsigned char* gbase) {
                asm volatile("s_mov_b32 m0, %0\n\ts_nop 0\n\tglobal_load_lds_dwordx4 %1, %2" :: "s"(lds), "v"(voff), "s"(gbase) : "memory");
            };
            auto stage = [&](int kk, int buf) {
                const unsigned lb = lds_base + buf * STAGE_BYTES + wave * 2048;
                const unsigned char* xk = xb + kk * BK;
                const unsigned char* wk = wb + kk * wstep;
                dma(lb, xoff[0], xk);
                dma(lb + 1024, xoff[1], xk);
                dma(lb + BM * BK, wo0, wk);
                dma(lb + BM * BK + 1024, wo1, wk);
            };

#define RD_X(R0, R1, A, OFF) "ds_read_b128 v[" #R0 ":" #R1 "], " A " offset:" #OFF "\n\t"
#define RD_W(R0, R1, A, OFF) "ds_read_b64_tr_b8 v[" #R0 ":" #R1 "], " A " offset:" #OFF "\n\t"
#define MM(C0, C1, W0, W1, X0, X1) "v_mfma_f32_16x16x128_f8f6f4 v[" #C0 ":" #C1 "], v[" #W0 ":" #W1 "], v[" #X0 ":" #X1 "], v[" #C0 ":" #C1 "] cbsz:1\n\t"
#define CLOB_ALL "v96","v97","v98","v99","v100","v101","v102","v103","v104","v105","v106","v107","v108","v109","v110","v111", \
               "v112","v113","v114","v115","v116","v117","v118","v119","v120","v121","v122","v123","v124","v125","v126","v127", \
               "v128","v129","v130","v131","v132","v133","v134","v135","v136","v137","v138","v139","v140","v141","v142","v143", \
               "v144","v145","v146","v147","v148","v149","v150","v151","v152","v153","v154","v155","v156","v157","v158","v159", \
               "v160","v161","v162","v163","v164","v165","v166","v167","v168","v169","v170","v171","v172","v173","v174","v175", \
               "v176","v177","v178","v179","v180","v181","v182","v183","v184","v185","v186","v187","v188","v189","v190","v191", \
               "v192","v193","v194","v195","v196","v197","v198","v199","v200","v201","v202","v203","v204","v205","v206","v207", \
               "v208","v209","v210","v211","v212","v213","v214","v215","v216","v217","v218","v219","v220","v221","v222","v223"
#define READ_A1 RD_W(160,161,"%2",0) RD_W(162,163,"%2",1024) RD_W(164,165,"%2",2048) RD_W(166,167,"%2",3072) \
                RD_X(128,131,"%0",0) RD_X(132,135,"%1",0) RD_X(136,139,"%0",2048) RD_X(140,143,"%1",2048)
#define READ_A2 RD_X(144,147,"%0",4096) RD_X(148,151,"%1",4096) RD_X(152,155,"%0",6144) RD_X(156,159,"%1",6144) \
                RD_W(168,169,"%3",0) RD_W(170,171,"%3",1024) RD_W(172,173,"%3",2048) RD_W(174,175,"%3",3072)
#define ZERO4(R) "v_mov_b32 v" #R ", 0\n\t"
            auto lds_addr = [&](int b, unsigned& a0, unsigned& a1, unsigned& a2, unsigned& a3) {
                const unsigned lb = lds_base + b * STAGE_BYTES;
                a0 = lb + xa[0]; a1 = lb + xa[1]; a2 = lb + wa[0]; a3 = lb + wa[1];
            };
            static_assert(NSTAGE == 5, "the K loop below prefetches exactly one slab (4 steps) ahead");
            slab(0);
#pragma unroll
            for (int i = 0; i < 4; ++i) stage(i, i);
            asm volatile(
                ZERO4(96) ZERO4(97) ZERO4(98) ZERO4(99) ZERO4(100) ZERO4(101) ZERO4(102) ZERO4(103)
                ZERO4(104) ZERO4(105) ZERO4(106) ZERO4(107) ZERO4(108) ZERO4(109) ZERO4(110) ZERO4(111)
                ZERO4(112) ZERO4(113) ZERO4(114) ZERO4(115) ZERO4(116) ZERO4(117) ZERO4(118) ZERO4(119)
                ZERO4(120) ZERO4(121) ZERO4(122) ZERO4(123) ZERO4(124) ZERO4(125) ZERO4(126) ZERO4(127)
                ::: CLOB_ALL);
            asm volatile("s_waitcnt vmcnt(12)" ::: "memory");
            __builtin_amdgcn_s_barrier();
            {
                unsigned a0, a1, a2, a3;
                lds_addr(0, a0, a1, a2, a3);
                asm volatile(READ_A1 READ_A2 :: "v"(a0), "v"(a1), "v"(a2), "v"(a3) : CLOB_ALL, "memory");
            }
            int buf = 0;
#define STEP(WAITN, STG, KK, CX0,CX0e,CX1,CX1e,CX2,CX2e,CX3,CX3e,CW0,CW0e,CW1,CW1e, \
             W0,W1,W2,W3,W4,W5,W6,W7, X0,X1,X2,X3,X4,X5,X6,X7,X8,X9,XA,XB,XC,XD,XE,XF, V0,V1,V2,V3,V4,V5,V6,V7) \
            do { \
                int nb = buf + 1; if (nb == NSTAGE) nb = 0; \
                unsigned a0, a1, a2, a3; \
                lds_addr(nb, a0, a1, a2, a3); \
                asm volatile("s_waitcnt vmcnt(" #WAITN ")" ::: "memory"); \
                __builtin_amdgcn_s_barrier(); \
                asm volatile("s_waitcnt lgkmcnt(0)\n\t" \
                    MM(96,99, CW0, CW0e, CX0, CX0e) RD_W(W0,W1,"%2",0) RD_W(W2,W3,"%2",1024) \
                    MM(100,103, CW0, CW0e, CX1, CX1e) RD_W(W4,W5,"%2",2048) RD_W(W6,W7,"%2",3072) \
                    MM(104,107, CW0, CW0e, CX2, CX2e) RD_X(X0,X1,"%0",0) RD_X(X2,X3,"%1",0) \
                    MM(108,111, CW0, CW0e, CX3, CX3e) RD_X(X4,X5,"%0",2048) RD_X(X6,X7,"%1",2048) \
                    :: "v"(a0), "v"(a1), "v"(a2) : CLOB_ALL, "memory"); \
                if (STG) { int pb = buf - 1; if (pb < 0) pb += NSTAGE; stage(KK, pb); } \
                asm volatile( \
                    MM(112,115, CW1, CW1e, CX0, CX0e) RD_X(X8,X9,"%0",4096) RD_X(XA,XB,"%1",4096) \
                    MM(116,119, CW1, CW1e, CX1, CX1e) RD_X(XC,XD,"%0",6144) RD_X(XE,XF,"%1",6144) \
                    MM(120,123, CW1, CW1e, CX2, CX2e) RD_W(V0,V1,"%2",0) RD_W(V2,V3,"%2",1024) \
                    MM(124,127, CW1, CW1e, CX3, CX3e) RD_W(V4,V5,"%2",2048) RD_W(V6,V7,"%2",3072) \
                    :: "v"(a0), "v"(a1), "v"(a3) : CLOB_ALL, "memory"); \
                buf = nb; \
            } while (0)
#define STEP_A(WAITN, STG, KK) STEP(WAITN, STG, KK, 128,135,136,143,144,151,152,159,160,167,168,175, \
                     208,209,210,211,212,213,214,215, 176,179,180,183,184,187,188,191,192,195,196,199,200,203,204,207, \
                     216,217,218,219,220,221,222,223)
#define STEP_B(WAITN, STG, KK) STEP(WAITN, STG, KK, 176,183,184,191,192,199,200,207,208,215,216,223, \
                     160,161,162,163,164,165,166,167, 128,131,132,135,136,139,140,143,144,147,148,151,152,155,156,159, \
                     168,169,170,171,172,173,174,175)
            for (int ps = 0; ps < NR - 1; ++ps) {
                slab(ps + 1);
                STEP_A(8, 1, 0);
                STEP_B(8, 1, 1);
                STEP_A(8, 1, 2);
                STEP_B(8, 1, 3);
            }
            STEP_A(8, 0, 0);
            STEP_B(4, 0, 1);
            STEP_A(0, 0, 2);
            asm volatile("s_waitcnt lgkmcnt(0)\n\t"
                MM(96,99, 208,215, 176,183) MM(100,103, 208,215, 184,191) MM(104,107, 208,215, 192,199) MM(108,111, 208,215, 200,207)
                MM(112,115, 216,223, 176,183) MM(116,119, 216,223, 184,191) MM(120,123, 216,223, 192,199) MM(124,127, 216,223, 200,207)
                "s_nop 15\n\ts_nop 15"
                ::: CLOB_ALL, "memory");
            f32x4 acc[2][4];
#define GET4(T, R0, R1, R2, R3) do { float y0, y1, y2, y3; \
                asm volatile("v_mul_f32 %0, %4, v" #R0 "\n\tv_mul_f32 %1, %4, v" #R1 "\n\tv_mul_f32 %2, %4, v" #R2 "\n\tv_mul_f32 %3, %4, v" #R3 \
                    : "=&v"(y0), "=&v"(y1), "=&v"(y2), "=&v"(y3) : "v"(scale) : CLOB_ALL); \
                T = f32x4{y0, y1, y2, y3}; } while (0)
            GET4(acc[0][0], 96, 97, 98, 99);     GET4(acc[0][1], 100, 101, 102, 103);
            GET4(acc[0][2], 104, 105, 106, 107); GET4(acc[0][3], 108, 109, 110, 111);
            GET4(acc[1][0], 112, 113, 114, 115); GET4(acc[1][1], 116, 117, 118, 119);
            GET4(acc[1][2], 120, 121, 122, 123); GET4(acc[1][3], 124, 125, 126, 127);

            const bool bad = __hip_atomic_load(tmo, __ATOMIC_RELAXED, __HIP_MEMORY_SCOPE_AGENT) == call;
            if (bad) {
                const float qnan = __builtin_bit_cast(float, 0x7fc00000u);
#pragma unroll
                for (int a = 0; a < 2; ++a)
#pragma unroll
                    for (int b = 0; b < 4; ++b) acc[a][b] = f32x4{qnan, qnan, qnan, qnan};
            }
            if (qq == rank) {
                const __amdgpu_buffer_rsrc_t ro = make_rsrc(p.out, (unsigned)((size_t)MS * NG * 4));
                const unsigned dst0 = ((unsigned)(j * BM + wm * 64 + t16) * NG + rank * NS + ct * BN + wn * 32 + g * 4) * 4;
#pragma unroll
                for (int mi = 0; mi < 4; ++mi)
#pragma unroll
                    for (int ni = 0; ni < 2; ++ni) {
                        f32x4 y;
#pragma unroll
                        for (int e = 0; e < 4; ++e) { const float v = acc[ni][mi][e]; y[e] = v < 0.f ? 0.f : v; }
                        __builtin_amdgcn_raw_buffer_store_b128(__builtin_bit_cast(u32x4, y), ro, dst0 + (mi * 16 * NG + ni * 16) * 4, 0, OUT_AUX);
                    }
            } else {
                __syncthreads();
                unsigned char* const wl = smem + wave * BLK_BYTES;
                const unsigned val0 = lds_base + wave * BLK_BYTES + 256;
                unsigned posb = val0, word = 0;
#pragma unroll
                for (int ni = 0; ni < 2; ++ni)
#pragma unroll
                    for (int mi = 0; mi < 4; ++mi)
#pragma unroll
                        for (int e = 0; e < 4; ++e) {
                            unsigned t, c;
                            unsigned long long co;
                            asm volatile(
                                "v_cmp_nge_f32 vcc, 0, %[v]\n\t"
                                "s_nop 1\n\t"
                                "v_addc_co_u32 %[w], %[co], %[w], %[w], vcc\n\t"
                                "v_mbcnt_lo_u32_b32 %[t], vcc_lo, 0\n\t"
                                "v_mbcnt_hi_u32_b32 %[t], vcc_hi, %[t]\n\t"
                                "v_lshl_add_u32 %[t], %[t], 2, %[p]\n\t"
                                "s_mov_b64 exec, vcc\n\t"
                                "ds_write_b32 %[t], %[v]\n\t"
                                "s_mov_b64 exec, -1\n\t"
                                "s_bcnt1_i32_b64 %[c], vcc\n\t"
                                "s_lshl2_add_u32 %[p], %[c], %[p]\n\t"
                                "s_nop 0"
                                : [t] "=&v"(t), [c] "=&s"(c), [co] "=&s"(co), [w] "+v"(word), [p] "+s"(posb)
                                : [v] "v"(acc[ni][mi][e])
                                : "vcc", "scc", "memory");
                        }
                const unsigned pos = (posb - val0) >> 2;
                *(unsigned*)(wl + lane * 4) = word;
                asm volatile("s_waitcnt lgkmcnt(0)" ::: "memory");
                const unsigned bytes = 256u + pos * 4u;
                const __amdgpu_buffer_rsrc_t rs = make_rsrc(wsme, (unsigned)WS_USED);
                const unsigned blk = (j * NCT + ct) * 8 + wave;
                const unsigned dst0 = (unsigned)(OFF_O + (size_t)qq * O_SLAB) + blk * BLK_BYTES;
#pragma unroll
                for (int c = 0; c < 9; ++c) {
                    const unsigned off = c * 1024 + lane * 16;
                    if (off < bytes)
                        __builtin_amdgcn_raw_buffer_store_b128(*(const u32x4*)(wl + off), rs, off, dst0, AUX_SYS);
                }
                WAIT_VM0();
                if (lane == 0)
                    __hip_atomic_store((unsigned long long*)(p.ws[qq] + OFF_FO) + rank * 512 + blk,
                                       ((unsigned long long)call << 32) | pos, __ATOMIC_RELAXED, __HIP_MEMORY_SCOPE_SYSTEM);
            }
            __syncthreads();
        }
    }
}

}

extern "C" void kernel_launch(int rank, int nranks, ncclComm_t comm,
                              hipStream_t stream,
                              void* const* d_in, const int* in_sizes, int n_in,
                              void* d_out, int out_size,
                              const int* in_dtypes, int out_dtype,
                              void* const* d_out_all,
                              void* const* d_ws_all, size_t ws_bytes) {
    (void)comm; (void)in_dtypes; (void)out_dtype; (void)d_out_all;
    if (nranks != NR || n_in < 4 || rank < 0 || rank >= NR || ws_bytes < WS_USED ||
        in_sizes[0] != MG * KS || in_sizes[1] != KS * NG || out_size != MS * NG) {
        fprintf(stderr, "kernel_launch: unexpected problem shape\n");
        return;
    }
    Params p;
    p.x = (const unsigned char*)d_in[0];
    p.w = (const unsigned char*)d_in[1];
    p.sx = (const float*)d_in[2];
    p.sw = (const float*)d_in[3];
    p.out = (float*)d_out;
    for (int i = 0; i < NR; ++i) p.ws[i] = (unsigned char*)d_ws_all[i];
    p.rank = rank;
    hipLaunchKernelGGL(gemm_rs_fused, dim3(NWG), dim3(512), 0, stream, p);
    HIP_CHECK(hipGetLastError());
}
```
